```python
import numpy as np

kernel_name = "gpu_gemm_ar_m4096_k4096_n4096_bf16_qe4m3"


ABSMAX_RTOL = 0.09

COMPUTE_DTYPE = "bfloat16"
EPILOGUE = "quant_e4m3"
IN_QUANT = None
OUT_QUANT = "e4m3"
IN_DTYPES = {"x": "bfloat16", "w_mat": "bfloat16"}
WORLD_SIZE = 8

def _to_bf16(x):
    u = np.ascontiguousarray(x, dtype=np.float32).view(np.uint32)
    return ((u + 0x7FFF + ((u >> 16) & 1)) & 0xFFFF0000).view(np.float32)

def setup_inputs():

    x = _to_bf16(np.random.randn(4096, 4096).astype(np.float32))
    w_mat = _to_bf16((np.random.randn(4096, 4096) / np.sqrt(4096)).astype(np.float32))
    return {"x": x, "w_mat": w_mat}

def _snap_fp8(x, mant_bits, max_abs, min_normal):
    a = np.abs(x.astype(np.float64))
    _m, _e = np.frexp(a)
    q = np.where(a >= min_normal, np.ldexp(1.0, _e - 1 - mant_bits),
                 min_normal * 2.0 ** -mant_bits)
    snapped = np.minimum(np.round(a / q) * q, max_abs)
    return (np.sign(x) * snapped).astype(np.float32)
def _quant_dequant(y, amax):
    scale = amax / 448.0
    q = _snap_fp8(y / scale, 3, 448.0, 2.0 ** -6)
    return (q * scale).astype(np.float32)
def _epilogue(y):
    return _quant_dequant(y, np.abs(y).max())

def reference(x, w_mat):
    y = (x @ w_mat).astype(np.float32)
    return _epilogue(y)
```

```cpp
#include <hip/hip_runtime.h>
#include <rccl/rccl.h>
#include <cstdio>
#include <cstdint>

typedef unsigned int u32;
typedef unsigned long long u64;
typedef u32 u32x4 __attribute__((ext_vector_type(4)));
typedef u32 u32x2 __attribute__((ext_vector_type(2)));
typedef __bf16 bf16x2 __attribute__((ext_vector_type(2)));
typedef float f32x2 __attribute__((ext_vector_type(2)));
typedef __amdgpu_buffer_rsrc_t rsrc_t;

#define AFF 1
#define NSJ 5
#define SW 2
#define WR (AFF ? (NSJ * 8 / 7) * SW : NSJ * 8 * SW / 7)
#define AGMODE 0
#define PDIRECT 0
#define HELP 0
#define ABL 0

constexpr u32 MiB = 1u << 20;
constexpr u32 PIECE = 8192;
constexpr u32 OFF_XS = 0;
constexpr u32 OFF_XO = 4 * MiB;
constexpr u32 OFF_W = 8 * MiB;
constexpr u32 OFF_P = 16 * MiB;
constexpr u32 OFF_PO = 20 * MiB;
constexpr u32 OFF_FX = 24 * MiB;
constexpr u32 OFF_FW = OFF_FX + 4096;
constexpr u32 OFF_FP = OFF_FW + 4096;
constexpr u32 OFF_G = OFF_FP + 8192;
constexpr u32 OFF_FD = OFF_G + 8192;
constexpr u32 OFF_CNT = OFF_FD + 8192;
constexpr u32 OFF_Q = OFF_CNT + 16384;
constexpr u32 OFF_DBG = OFF_Q + 1024;
constexpr u32 WS_USED = OFF_DBG + 256 * 128;

constexpr u32 SLOT = 4 * PIECE;
constexpr u32 CTRL = 4 * SLOT;
constexpr u32 LDS_BYTES = CTRL + 256;
constexpr u32 C_FULL = CTRL + 0;
constexpr u32 C_CTL = CTRL + 16;
constexpr u32 C_CONS = CTRL + 32;
constexpr u32 C_WMAX = CTRL + 96;
constexpr u32 C_GMAX = CTRL + 128;
static_assert(C_FULL == 0x20000 && SLOT == 0x8000, "literals in the asm loops");
constexpr u64 DEADLINE = 50000000ull;
constexpr int AUX_SYS = 17;

struct Params {
  const void* x;
  const void* w;
  char* ws[8];
  unsigned char* out[8];
  int rank;
};

__device__ __forceinline__ u32 rfl(u32 v) { return __builtin_amdgcn_readfirstlane(v); }
__device__ __forceinline__ rsrc_t mk_rsrc(const void* p, u32 bytes) {
  return __builtin_amdgcn_make_buffer_rsrc((void*)p, (short)0, (int)bytes, 0x00020000);
}
__device__ __forceinline__ void lds_st32(u32 addr, u32 v) { asm volatile("ds_write_b32 %0, %1" ::"v"(addr), "v"(v) : "memory"); }
__device__ __forceinline__ void st128(u32x4 v, rsrc_t r, u32 voff, u32 soff) {
  asm volatile("buffer_store_dwordx4 %0, %1, %2, %3 offen sc0 sc1\n\ts_nop 1" ::"v"(v), "v"(voff), "s"(r), "s"(soff) : "memory");
}
template <int OFF> __device__ __forceinline__ void st128i(u32x4 v, rsrc_t r, u32 voff, u32 soff) {
  asm volatile("buffer_store_dwordx4 %0, %1, %2, %3 offen offset:%4 sc0 sc1\n\ts_nop 1" ::"v"(v), "v"(voff), "s"(r), "s"(soff), "i"(OFF) : "memory");
}
__device__ __forceinline__ void sgpr_pad(u32 s) { asm volatile("s_nop 4" ::"s"(s)); }
#define STAMP(i) do { } while (0)

__device__ __forceinline__ u32 row0_of(u32 r, u32 u, u32 own) {
  const u32 l = r >> 2, i = (r >> 1) & 1;
  const u32 quad = (l + 2 * i + (u >> 3)) & 3;
  const u32 half = own ? l : 1 - l;
  return quad * 1024 + half * 512 + (u & 7) * 64;
}
__device__ __forceinline__ u32 col0_of(u32 r, u32 v) {
  const u32 l = r >> 2, j = r & 1;
  return ((l + 2 * j + (v >> 3)) & 3) * 1024 + (v & 7) * 128;
}

struct PieceD { u32 kind, src, dst, flag, dep; };
__device__ __forceinline__ PieceD decode(u32 q, u32 rank, u32 dest, u32 nk, u32 kbase, u32 step0) {
  PieceD d;
  const u32 nx = 16 * nk;
  d.dep = OFF_FP;
  if (q >= 4 * nx) {
    const u32 qq = q - 4 * nx, l = rank >> 2, l2 = 1 - l;
    const u32 qa0 = (l + (rank & 2) - l2) & 3, qb0 = (l + 2 * (rank & 1) - l2) & 3;
    const u32 uh = (qa0 >> 1) != ((dest >> 1) & 1), vh = (qb0 >> 1) != (dest & 1);
    const u32 qa = (qa0 + uh) & 3, qb = (qb0 + vh) & 3;
    const u32 ul = (qq >> 2) & 3, vl = qq >> 4, h = qq & 3;
    const u32 tile = (uh * 4 + ul) * 16 + vh * 8 + vl, ptile = ((qa & 1) * 4 + ul) * 16 + (qb & 1) * 8 + vl;
    d.kind = 2;
    d.src = OFF_PO + tile * 32768 + h * PIECE;
    d.dst = OFF_P + ptile * 32768 + h * PIECE;
    d.flag = OFF_FP + ptile * 64 + h * 4;
    d.dep = OFF_FP + tile * 64 + 32 + h * 4;
  } else if (q >= nx && q < 3 * nx) {
    const u32 qq = q - nx;
    const u32 o = (nk == 5) ? qq / 10 : qq / 6, rem = qq - o * 2 * nk;
    const u32 ks = rem >> 1, half = rem & 1, v = (o >> 1) + 8 * (o & 1), step = step0 + ks;
    d.kind = 1;
    d.src = (kbase + 64 * ks) * 8192 + (col0_of(dest, v) + 64 * half) * 2;
    d.dst = OFF_W + ((v * 32 + step) * 2 + half) * PIECE;
    d.flag = OFF_FW + ((2 * v + half) * 32 + step) * 4;
  } else {
    const u32 own = q >= nx, qq = own ? q - 3 * nx : q;
    const u32 ks = qq >> 4, u = qq & 15, step = step0 + ks;
    d.kind = 0;
    d.src = row0_of(dest, u, own) * 1024 + (kbase + 64 * ks) * 2;
    d.dst = (own ? OFF_XO : OFF_XS) + (u * 32 + step) * PIECE;
    d.flag = OFF_FX + ((own * 16 + u) * 32 + step) * 4;
  }
  d.kind = rfl(d.kind); d.src = rfl(d.src); d.dst = rfl(d.dst); d.flag = rfl(d.flag); d.dep = rfl(d.dep);
  return d;
}

__device__ __forceinline__ void piece_load(u32x4 (&v)[8], const PieceD& d, rsrc_t rx, rsrc_t rw, rsrc_t rl, u32 lane) {
  const u32 kind = rfl(d.kind);
  const u32 vo = (lane >> 3) * (kind == 1 ? 65536u : kind == 0 ? 1024u : 128u) + (lane & 7) * 16;
  const rsrc_t r = kind == 1 ? rw : kind == 0 ? rx : rl;
  const u32 st = rfl(kind == 2 ? 1024u : 8192u);
  u32 so = rfl(d.src);
  asm volatile("s_nop 4\n\tbuffer_load_dwordx4 %0, %9, %10, %8 offen sc1\n\ts_add_u32 %8, %8, %11\n\t"
               "buffer_load_dwordx4 %1, %9, %10, %8 offen sc1\n\ts_add_u32 %8, %8, %11\n\t"
               "buffer_load_dwordx4 %2, %9, %10, %8 offen sc1\n\ts_add_u32 %8, %8, %11\n\t"
               "buffer_load_dwordx4 %3, %9, %10, %8 offen sc1\n\ts_add_u32 %8, %8, %11\n\t"
               "buffer_load_dwordx4 %4, %9, %10, %8 offen sc1\n\ts_add_u32 %8, %8, %11\n\t"
               "buffer_load_dwordx4 %5, %9, %10, %8 offen sc1\n\ts_add_u32 %8, %8, %11\n\t"
               "buffer_load_dwordx4 %6, %9, %10, %8 offen sc1\n\ts_add_u32 %8, %8, %11\n\t"
               "buffer_load_dwordx4 %7, %9, %10, %8 offen sc1"
               : "=&v"(v[0]), "=&v"(v[1]), "=&v"(v[2]), "=&v"(v[3]), "=&v"(v[4]), "=&v"(v[5]), "=&v"(v[6]), "=&v"(v[7]), "+&s"(so)
               : "v"(vo), "s"(r), "s"(st) : "scc", "memory");
}
__device__ __forceinline__ void dep_load(u32& f, u32 dep, rsrc_t rl) {
  asm volatile("s_nop 4\n\tbuffer_load_dword %0, off, %1, %2 sc0 sc1" : "=&v"(f) : "s"(rl), "s"(rfl(dep)) : "memory");
}
__device__ __forceinline__ void claim(u32& c, u32 qoff, rsrc_t rl) {
  c = 1;
  asm volatile("s_mov_b64 exec, 1\n\tbuffer_atomic_add %0, off, %1, %2 sc0\n\ts_mov_b64 exec, -1" : "+v"(c) : "s"(rl), "s"(qoff) : "memory");
}
#define WAITV(N, v, f, c) asm volatile("s_waitcnt vmcnt(" #N ")" : "+v"(v[0]), "+v"(v[1]), "+v"(v[2]), "+v"(v[3]), "+v"(v[4]), "+v"(v[5]), "+v"(v[6]), "+v"(v[7]), "+v"(f), "+v"(c) :: "memory")
template <int E> __device__ __forceinline__ void w_store(const u32x4 (&v)[8], rsrc_t rd, u32 wo, u32 dst) {
  u32x4 o;
#pragma unroll
  for (int j = 0; j < 4; ++j)
    o[j] = __builtin_amdgcn_perm(v[2 * j + 1][E >> 1], v[2 * j][E >> 1], (E & 1) ? 0x07060302u : 0x05040100u);
  st128i<((E & 3) + 8 * (E >> 2)) * 128>(o, rd, wo ^ ((((E >> 1) & 1) + 4 * (E >> 2)) << 4), dst);
}
__device__ __forceinline__ void piece_store(const u32x4 (&v)[8], const PieceD& d, rsrc_t rd, u32 lane) {
  const u32 hi = lane >> 3, lo = lane & 7, dst = rfl(d.dst), kind = rfl(d.kind);
  sgpr_pad(dst);
  if (kind == 1) {
    const u32 wo = (32 * (lo >> 2) + 16 * (lo & 1) + 4 * ((lo >> 1) & 1)) * 128 + ((hi ^ (lo & 2)) << 4);
    w_store<0>(v, rd, wo, dst); w_store<1>(v, rd, wo, dst); w_store<2>(v, rd, wo, dst); w_store<3>(v, rd, wo, dst);
    w_store<4>(v, rd, wo, dst); w_store<5>(v, rd, wo, dst); w_store<6>(v, rd, wo, dst); w_store<7>(v, rd, wo, dst);
  } else {
    const u32 xe = hi * 128 + ((lo ^ (hi >> 1)) << 4), xo = xe ^ 64, dst2 = dst + 4096;
    st128i<0>(v[0], rd, xe, dst); st128i<1024>(v[1], rd, xo, dst); st128i<2048>(v[2], rd, xe, dst); st128i<3072>(v[3], rd, xo, dst);
    st128i<0>(v[4], rd, xe, dst2); st128i<1024>(v[5], rd, xo, dst2); st128i<2048>(v[6], rd, xe, dst2); st128i<3072>(v[7], rd, xo, dst2);
  }
}

template <bool DYN>
__device__ __forceinline__ void send_stream(const Params& p, u32 dest, u32 q0, u32 qs, u32 seqv, u32 lane, u64 t0) {
  const u32 rank = p.rank;
  const bool same = (dest >> 2) == (rank >> 2);
  const u32 nk = same ? 5 : 3, kbase = same ? 0 : 320, step0 = same ? (rank & 3) * 5 : 20 + (rank & 3) * 3;
  const u32 total = (ABL == 1 || PDIRECT) ? 64 * nk : 320;
  if (q0 >= total) return;
  const rsrc_t rx = mk_rsrc(p.x, 4 * MiB), rw = mk_rsrc(p.w, 4 * MiB), rd = mk_rsrc(p.ws[dest], WS_USED), rl = mk_rsrc(p.ws[rank], WS_USED);
  u32x4 bufA[8], bufB[8];
  u32 f = 0;
  u32 cl = 0, i1 = q0 + qs, i2 = q0 + 2 * qs;
  PieceD dc = decode(q0, rank, dest, nk, kbase, step0);
  piece_load(bufA, dc, rx, rw, rl, lane);
  PieceD dn = decode(min(i1, total - 1), rank, dest, nk, kbase, step0);
  WAITV(0, bufA, f, cl);
  const u32 seq = (rfl(seqv) + 1);
  const u32 qoff = OFF_Q + ((seq & 1) * 8 + dest) * 64;
  u32 prevflag = 0;
  bool rdyn = true;
#define FLAG(off) __builtin_amdgcn_raw_buffer_store_b32(seq, rd, 0, rfl(off), AUX_SYS)
#define SEND_HALF(cur, nxt)                                                                   \
  {                                                                                           \
    const bool more = i1 < total;                                                             \
    const PieceD d2 = decode(min(i2, total - 1), rank, dest, nk, kbase, step0);               \
    const u32 slow = rfl(more && !rdyn);                                                      \
    if (slow) {                                                                               \
      piece_store(cur, dc, rd, lane);                                                         \
      asm volatile("s_waitcnt vmcnt(0)" ::: "memory");                                        \
      if (lane == 0) { if (prevflag) FLAG(prevflag); FLAG(dc.flag); }                         \
      prevflag = 0;                                                                           \
      for (;;) {                                                                              \
        u32 g;                                                                                \
        dep_load(g, dn.dep, rl);                                                              \
        asm volatile("s_waitcnt vmcnt(0)" : "+v"(g) :: "memory");                             \
        if (rfl(g) >= seq || __builtin_amdgcn_s_memrealtime() - t0 > DEADLINE) break; \
        __builtin_amdgcn_s_sleep(4);                                                          \
      }                                                                                       \
    }                                                                                         \
    if (DYN) claim(cl, qoff, rl);                                                             \
    dep_load(f, d2.dep, rl);                                                                  \
    piece_load(nxt, dn, rx, rw, rl, lane);                 \
    if (!slow) piece_store(cur, dc, rd, lane);                                                \
    else asm volatile("s_waitcnt vmcnt(0)" ::: "memory");                                     \
    WAITV(8, nxt, f, cl);                                                                      \
    if (prevflag && lane == 0) FLAG(prevflag);                                                \
    prevflag = slow ? 0u : dc.flag;                                                           \
    if (!more) break;                                                                         \
    rdyn = d2.kind != 2 || rfl(f) >= seq;                                       \
    dc = dn; dn = d2;                                                                         \
    i1 = i2; i2 = DYN ? 3 * qs + rfl(cl) : i2 + qs;                                           \
  }
  for (;;) {
    SEND_HALF(bufA, bufB)
    SEND_HALF(bufB, bufA)
  }
  asm volatile("s_waitcnt vmcnt(0)" ::: "memory");
  if (prevflag && lane == 0) __builtin_amdgcn_raw_buffer_store_b32(seq, rd, 0, rfl(prevflag), AUX_SYS);
}

#define GL4(so) "buffer_load_dwordx4 %[voff], %[rs], " so " offen sc1 lds\n\t"                 \
                "buffer_load_dwordx4 %[voff], %[rs], " so " offen offset:1024 sc1 lds\n\t"     \
                "buffer_load_dwordx4 %[voff], %[rs], " so " offen offset:2048 sc1 lds\n\t"     \
                "buffer_load_dwordx4 %[voff], %[rs], " so " offen offset:3072 sc1 lds\n\t"
#define NEXT4(so) "s_add_u32 m0, m0, 0x1000\n\ts_add_u32 " so ", " so ", 0x1000\n\t"
__device__ __forceinline__ bool load_job(u32 lw, u32 isB, u32 U, u32 v, u32& n, char* myws, u32 lane, u32 seq, u64 t0) {
  const rsrc_t rs = mk_rsrc(myws, WS_USED);
  const u32* fx = (const u32*)(myws + OFF_FX) + (isB * 16 + 2 * U) * 32 + lane;
  const u32* fw = (const u32*)(myws + OFF_FW) + (2 * v) * 32 + lane;
  const u32* fp = (const u32*)(myws + OFF_FP + (U * 16 + v) * 64) + (lane & 3);
  const u32 xbase = (isB ? OFF_XO : OFF_XS) + 2 * U * 32 * PIECE, wbase = OFF_W + v * 64 * PIECE;
  u32 rem = 0x11111111u << lw, rdy = 0, needP = isB && lw == 0;
  while (rem | needP) {
    const u32 cand = rdy & rem;
    if (!cand) {
      const u32 a = __hip_atomic_load(fx, __ATOMIC_RELAXED, __HIP_MEMORY_SCOPE_SYSTEM);
      const u32 c = __hip_atomic_load(fw, __ATOMIC_RELAXED, __HIP_MEMORY_SCOPE_SYSTEM);
      if (needP && __all(ABL == 4 || __hip_atomic_load(fp, __ATOMIC_RELAXED, __HIP_MEMORY_SCOPE_SYSTEM) >= seq)) {
        asm volatile("ds_or_b32 %0, %1" ::"v"(C_CTL), "v"(lane == 0 ? 2u : 0u) : "memory");
        needP = 0;
      }
      const u64 m = __ballot(ABL == 4 || (a >= seq && c >= seq));
      rdy = (u32)m & (u32)(m >> 32);
      if (__builtin_amdgcn_s_memrealtime() - t0 > DEADLINE) return false;
      continue;
    }
    const u32 step = __builtin_ctz(cand);
    rem &= ~(1u << step);
    u32 x = xbase + step * PIECE, w = wbase + step * 2 * PIECE, tmp, spin = 0;
    ++n;
    asm volatile(
        "1:\n\t"
        "ds_read_b128 v[52:55], %[acons]\n\t"
        "s_waitcnt lgkmcnt(0)\n\t"
        "v_min_u32 v52, v52, v53\n\t"
        "v_min3_u32 v52, v52, v54, v55\n\t"
        "s_add_u32 %[spin], %[spin], 1\n\t"
        "v_readfirstlane_b32 %[tmp], v52\n\t"
        "s_add_u32 %[tmp], %[tmp], 1\n\t"
        "s_cmp_eq_u32 %[tmp], %[n]\n\t"
        "s_cbranch_scc1 2f\n\t"
        "s_bitcmp0_b32 %[spin], 23\n\t"
        "s_cbranch_scc1 1b\n\t"
        "s_branch 3f\n\t"
        "2:\n\t"
        "s_mov_b32 m0, %[m0]\n\t"
        "s_mov_b32 %[spin], 0\n\t"
        GL4("%[x]") NEXT4("%[x]") GL4("%[x]")
        "s_add_u32 m0, m0, 0x1000\n\ts_add_u32 %[x], %[x], 0x3f000\n\t"
        GL4("%[x]") NEXT4("%[x]") GL4("%[x]")
        "s_add_u32 m0, m0, 0x1000\n\ts_nop 0\n\t"
        GL4("%[w]") NEXT4("%[w]") GL4("%[w]") NEXT4("%[w]") GL4("%[w]") NEXT4("%[w]") GL4("%[w]")
        "v_mov_b32 v52, %[n]\n\t"
        "s_waitcnt vmcnt(0)\n\t"
        "ds_write_b32 %[afull], v52\n\t"
        "3:"
        : [x] "+&s"(x), [w] "+&s"(w), [tmp] "=&s"(tmp), [spin] "+&s"(spin)
        : [voff] "v"(lane * 16), [rs] "s"(rs), [acons] "v"(C_CONS + lw * 16), [afull] "v"(C_FULL + lw * 4), [m0] "s"(lw * SLOT), [n] "s"(n)
        : "scc", "memory", "v52", "v53", "v54", "v55");
    if (spin) return false;
  }
  return true;
}

#define TRY(k, n) "s_cmp_gt_u32 %[f" #k "], " n "\n\ts_cbranch_scc1 2" #k "f\n\t"
#define TAKE(k, n, base)                                         \
      "2" #k ":\n\t"                                             \
      "s_add_u32 " n ", " n ", 1\n\t"                            \
      "s_add_u32 %[xb], %[xoff], " base "\n\t"                   \
      "s_add_u32 %[wb], %[woff], " base "\n\t"                   \
      "v_add_u32 v50, " #k "*16, %[ac]\n\t"                      \
      "v_mov_b32 v51, " n "\n\t"
#define RDSTATE "ds_read_b128 v[52:55], v48\n\tds_read_b32 v49, v48 offset:16\n\t"

__global__ void __launch_bounds__(512, 2) gemm_ar_q8_fused(Params p) {
  extern __shared__ __attribute__((aligned(16))) char lds[];
  const u32 lane = threadIdx.x & 63, wid = rfl(threadIdx.x >> 6), b = blockIdx.x, rank = p.rank;
  if ((u32)(size_t)lds != 0) __builtin_trap();
  char* const myws = p.ws[rank];
  const u64 t0 = __builtin_amdgcn_s_memrealtime();
  const u32 seqv = __hip_atomic_load((u32*)(myws + OFF_CNT + b * 64), __ATOMIC_RELAXED, __HIP_MEMORY_SCOPE_AGENT);
  const u32 xg = b & 7, jb = b >> 3;
  if (wid == 0) STAMP(0);
  if (jb < NSJ) {
    const u32 sb = jb * 8 + xg, g = sb * SW + wid, d = AFF ? sb % 7 : g % 7, j = AFF ? (sb / 7) * SW + wid : g / 7;
    if (wid < SW && j < WR && ABL != 4) send_stream<true>(p, (rank + 1 + d) & 7, j, WR, seqv, lane, t0);
    const u32 seq = rfl(seqv) + 1;
    __syncthreads();
    if (threadIdx.x == 0) __hip_atomic_store((u32*)(myws + OFF_CNT + b * 64), seq, __ATOMIC_RELAXED, __HIP_MEMORY_SCOPE_AGENT);
    if (b == 0 && threadIdx.x < 8)
      __hip_atomic_store((u32*)(myws + OFF_Q + ((~seq & 1) * 8 + threadIdx.x) * 64), 0u, __ATOMIC_RELAXED, __HIP_MEMORY_SCOPE_AGENT);
    return;
  }
  const u32 hasA = jb < NSJ + 16, hasB = !hasA || jb < 2 * NSJ;
  const u32 ia = jb - NSJ, ib = hasA ? 16 - NSJ + ia : jb - NSJ - 16;
  const u32 UA = (xg & 1) * 4 + (ia & 3), oA = (xg >> 1) + 4 * (ia >> 2), vA = (oA >> 1) + 8 * (oA & 1);
  const u32 UB = (xg & 1) * 4 + (ib & 3), oB = (xg >> 1) + 4 * (ib >> 2), vB = (oB >> 1) + 8 * (oB & 1);
  const u32 tileA = UA * 16 + vA, tileB = UB * 16 + vB;
  if (wid == 0) lds_st32(CTRL + lane * 4, 0);
  asm volatile("s_waitcnt lgkmcnt(0)\n\ts_barrier" ::: "memory");
  const u32 wm = wid & 1, wn = (wid >> 1) & 1;
  float y[64];
#pragma unroll
  for (int i = 0; i < 64; ++i) y[i] = 0.f;

  if (wid < 4) {
   if (ABL != 1) {
    asm volatile("v_accvgpr_write_b32 a0, 0\n\tv_accvgpr_write_b32 a1, 0\n\tv_accvgpr_write_b32 a2, 0\n\tv_accvgpr_write_b32 a3, 0\n\tv_accvgpr_write_b32 a4, 0\n\tv_accvgpr_write_b32 a5, 0\n\tv_accvgpr_write_b32 a6, 0\n\tv_accvgpr_write_b32 a7, 0\n\tv_accvgpr_write_b32 a8, 0\n\tv_accvgpr_write_b32 a9, 0\n\tv_accvgpr_write_b32 a10, 0\n\tv_accvgpr_write_b32 a11, 0\n\tv_accvgpr_write_b32 a12, 0\n\tv_accvgpr_write_b32 a13, 0\n\tv_accvgpr_write_b32 a14, 0\n\tv_accvgpr_write_b32 a15, 0\n\tv_accvgpr_write_b32 a16, 0\n\tv_accvgpr_write_b32 a17, 0\n\tv_accvgpr_write_b32 a18, 0\n\tv_accvgpr_write_b32 a19, 0\n\tv_accvgpr_write_b32 a20, 0\n\tv_accvgpr_write_b32 a21, 0\n\tv_accvgpr_write_b32 a22, 0\n\tv_accvgpr_write_b32 a23, 0\n\tv_accvgpr_write_b32 a24, 0\n\tv_accvgpr_write_b32 a25, 0\n\tv_accvgpr_write_b32 a26, 0\n\tv_accvgpr_write_b32 a27, 0\n\tv_accvgpr_write_b32 a28, 0\n\tv_accvgpr_write_b32 a29, 0\n\tv_accvgpr_write_b32 a30, 0\n\tv_accvgpr_write_b32 a31, 0\n\tv_accvgpr_write_b32 a32, 0\n\tv_accvgpr_write_b32 a33, 0\n\tv_accvgpr_write_b32 a34, 0\n\tv_accvgpr_write_b32 a35, 0\n\tv_accvgpr_write_b32 a36, 0\n\tv_accvgpr_write_b32 a37, 0\n\tv_accvgpr_write_b32 a38, 0\n\tv_accvgpr_write_b32 a39, 0\n\tv_accvgpr_write_b32 a40, 0\n\tv_accvgpr_write_b32 a41, 0\n\tv_accvgpr_write_b32 a42, 0\n\tv_accvgpr_write_b32 a43, 0\n\tv_accvgpr_write_b32 a44, 0\n\tv_accvgpr_write_b32 a45, 0\n\tv_accvgpr_write_b32 a46, 0\n\tv_accvgpr_write_b32 a47, 0\n\tv_accvgpr_write_b32 a48, 0\n\tv_accvgpr_write_b32 a49, 0\n\tv_accvgpr_write_b32 a50, 0\n\tv_accvgpr_write_b32 a51, 0\n\tv_accvgpr_write_b32 a52, 0\n\tv_accvgpr_write_b32 a53, 0\n\tv_accvgpr_write_b32 a54, 0\n\tv_accvgpr_write_b32 a55, 0\n\tv_accvgpr_write_b32 a56, 0\n\tv_accvgpr_write_b32 a57, 0\n\tv_accvgpr_write_b32 a58, 0\n\tv_accvgpr_write_b32 a59, 0\n\tv_accvgpr_write_b32 a60, 0\n\tv_accvgpr_write_b32 a61, 0\n\tv_accvgpr_write_b32 a62, 0\n\tv_accvgpr_write_b32 a63, 0" ::: "a0", "a1", "a2", "a3", "a4", "a5", "a6", "a7", "a8", "a9", "a10", "a11", "a12", "a13", "a14", "a15", "a16", "a17", "a18", "a19", "a20", "a21", "a22", "a23", "a24", "a25", "a26", "a27", "a28", "a29", "a30", "a31", "a32", "a33", "a34", "a35", "a36", "a37", "a38", "a39", "a40", "a41", "a42", "a43", "a44", "a45", "a46", "a47", "a48", "a49", "a50", "a51", "a52", "a53", "a54", "a55", "a56", "a57", "a58", "a59", "a60", "a61", "a62", "a63");
    u32 fo[4];
    {
      const u32 i = lane & 31, h = lane >> 5;
#pragma unroll
      for (int s = 0; s < 4; ++s) fo[s] = i * 128 + (((2 * s + h) ^ ((i >> 1) & 7)) << 4);
    }
    const rsrc_t rs = mk_rsrc(myws, WS_USED);
    const u32 seq = rfl(seqv) + 1;
    const u32 l = rank >> 2, l2 = 1 - l;
    const u32 qa = (l + (rank & 2) + (UA >> 2) - l2) & 3, qb = (l + 2 * (rank & 1) + (vA >> 3) - l2) & 3;
    const u32 prank = PDIRECT ? l2 * 4 + (qa >> 1) * 2 + (qb >> 1) : rank;
    const u32 ptile = ((qa & 1) * 4 + (UA & 3)) * 16 + (qb & 1) * 8 + (vA & 7);
    const rsrc_t rp = mk_rsrc(p.ws[prank], WS_USED);
    const u32 pdst = (PDIRECT ? OFF_P + ptile * 32768 : OFF_PO + tileA * 32768) + wid * PIECE;
    const u32 pflag = PDIRECT ? OFF_FP + ptile * 64 + wid * 4 : OFF_FP + tileA * 64 + 32 + wid * 4;
    u32 n0 = 0, n1 = 0, n2 = 0, n3 = 0, done = 0;
    const u32 total = 32 * (hasA + hasB);
    u32 want = hasB ? 3 : 1, stop = 32, inA = hasA;
    for (;;) {
      u32 f0, f1, f2, f3, ctl, xb, wb;
      asm volatile(
      "v_mov_b32 v48, 0x20000\n\t"
      RDSTATE
      "1:\n\t"
      "s_waitcnt lgkmcnt(0)\n\t"
      "v_readfirstlane_b32 %[ctl], v49\n\t"
      "v_readfirstlane_b32 %[f0], v52\n\t"
      "v_readfirstlane_b32 %[f1], v53\n\t"
      "v_readfirstlane_b32 %[f2], v54\n\t"
      "v_readfirstlane_b32 %[f3], v55\n\t"
      "s_and_b32 %[xb], %[ctl], %[want]\n\t"
      "s_cbranch_scc1 9f\n\t"
      TRY(0, "%[n0]") TRY(1, "%[n1]") TRY(2, "%[n2]") TRY(3, "%[n3]")
      "s_sleep 1\n\t"
      RDSTATE
      "s_branch 1b\n\t"
      TAKE(0, "%[n0]", "0") "s_branch 6f\n\t"
      TAKE(1, "%[n1]", "0x8000") "s_branch 6f\n\t"
      TAKE(2, "%[n2]", "0x10000") "s_branch 6f\n\t"
      TAKE(3, "%[n3]", "0x18000")
      "6:\n\t"
v_accvgpr_read_b32 %0, a80\n\tv_accvgpr_read_b32 %1, a81\n\tv_accvgpr_read_b32 %2, a82\n\tv_accvgpr_read_b32 %3, a83\n\tv_accvgpr_read_b32 %4, a84\n\tv_accvgpr_read_b32 %5, a85\n\tv_accvgpr_read_b32 %6, a86\n\tv_accvgpr_read_b32 %7, a87\n\tv_accvgpr_read_b32 %8, a88\n\tv_accvgpr_read_b32 %9, a89\n\tv_accvgpr_read_b32 %10, a90\n\tv_accvgpr_read_b32 %11, a91\n\tv_accvgpr_read_b32 %12, a92\n\tv_accvgpr_read_b32 %13, a93\n\tv_accvgpr_read_b32 %14, a94\n\tv_accvgpr_read_b32 %15, a95      "s_add_u32 %[done], %[done], 1\n\t"
      "s_cmp_lt_u32 %[done], %[stop]\n\t"
      "s_cbranch_scc1 1b\n\t"
      "9:\n\t"
      "s_waitcnt lgkmcnt(0)\n\t"
      "s_nop 15"
      : [n0] "+&s"(n0), [n1] "+&s"(n1), [n2] "+&s"(n2), [n3] "+&s"(n3), [done] "+&s"(done),
        [f0] "=&s"(f0), [f1] "=&s"(f1), [f2] "=&s"(f2), [f3] "=&s"(f3), [ctl] "=&s"(ctl), [xb] "=&s"(xb), [wb] "=&s"(wb)
      : [fo0] "v"(fo[0]), [fo1] "v"(fo[1]), [fo2] "v"(fo[2]), [fo3] "v"(fo[3]), [ac] "v"(C_CONS + wid * 4),
        [want] "s"(want), [stop] "s"(stop), [xoff] "s"(wm * PIECE), [woff] "s"(2 * PIECE + wn * PIECE)
      : "scc", "memory", "v48", "v49", "v50", "v51", "v52", "v53", "v54", "v55", "v56", "v57", "v58", "v59", "v60", "v61", "v62", "v63", "v64", "v65", "v66", "v67", "v68", "v69", "v70", "v71", "v72", "v73", "v74", "v75", "v76", "v77", "v78", "v79", "v80", "v81", "v82", "v83", "v84", "v85", "v86", "v87", "v88", "v89", "v90", "v91", "v92", "v93", "v94", "v95", "v96", "v97", "v98", "v99", "v100", "v101", "v102", "v103", "v104", "v105", "v106", "v107", "v108", "v109", "v110", "v111", "v112", "v113", "v114", "v115", "v116", "v117", "v118", "v119", "v120", "v121", "v122", "v123", "v124", "v125", "v126", "v127", "a0", "a1", "a2", "a3", "a4", "a5", "a6", "a7", "a8", "a9", "a10", "a11", "a12", "a13", "a14", "a15", "a16", "a17", "a18", "a19", "a20", "a21", "a22", "a23", "a24", "a25", "a26", "a27", "a28", "a29", "a30", "a31", "a32", "a33", "a34", "a35", "a36", "a37", "a38", "a39", "a40", "a41", "a42", "a43", "a44", "a45", "a46", "a47", "a48", "a49", "a50", "a51", "a52", "a53", "a54", "a55", "a56", "a57", "a58", "a59", "a60", "a61", "a62", "a63");
      if (ctl & 1) break;
      if (inA && done == 32) {
#define PACK_STORE(J0)                                                                              \
        {                                                                                           \
          _Pragma("unroll") for (int j = 0; j < 2; ++j) {                                           \
            u32x4 o4;                                                                               \
            _Pragma("unroll") for (int e = 0; e < 4; ++e) {                                         \
              f32x2 f = {d[8 * j + 2 * e], d[8 * j + 2 * e + 1]};                                   \
              o4[e] = __builtin_bit_cast(u32, __builtin_convertvector(f, bf16x2));                  \
            }                                                                                       \
            st128(o4, rp, lane * 16, pdst + (J0 + j) * 1024);                                       \
          }                                                                                         \
        }
        { float d[16]; asm volatile("v_accvgpr_read_b32 %0, a0\n\tv_accvgpr_read_b32 %1, a1\n\tv_accvgpr_read_b32 %2, a2\n\tv_accvgpr_read_b32 %3, a3\n\tv_accvgpr_read_b32 %4, a4\n\tv_accvgpr_read_b32 %5, a5\n\tv_accvgpr_read_b32 %6, a6\n\tv_accvgpr_read_b32 %7, a7\n\tv_accvgpr_read_b32 %8, a8\n\tv_accvgpr_read_b32 %9, a9\n\tv_accvgpr_read_b32 %10, a10\n\tv_accvgpr_read_b32 %11, a11\n\tv_accvgpr_read_b32 %12, a12\n\tv_accvgpr_read_b32 %13, a13\n\tv_accvgpr_read_b32 %14, a14\n\tv_accvgpr_read_b32 %15, a15" : "=v"(d[0]), "=v"(d[1]), "=v"(d[2]), "=v"(d[3]), "=v"(d[4]), "=v"(d[5]), "=v"(d[6]), "=v"(d[7]), "=v"(d[8]), "=v"(d[9]), "=v"(d[10]), "=v"(d[11]), "=v"(d[12]), "=v"(d[13]), "=v"(d[14]), "=v"(d[15])); PACK_STORE(0) }
        { float d[16]; asm volatile("v_accvgpr_read_b32 %0, a16\n\tv_accvgpr_read_b32 %1, a17\n\tv_accvgpr_read_b32 %2, a18\n\tv_accvgpr_read_b32 %3, a19\n\tv_accvgpr_read_b32 %4, a20\n\tv_accvgpr_read_b32 %5, a21\n\tv_accvgpr_read_b32 %6, a22\n\tv_accvgpr_read_b32 %7, a23\n\tv_accvgpr_read_b32 %8, a24\n\tv_accvgpr_read_b32 %9, a25\n\tv_accvgpr_read_b32 %10, a26\n\tv_accvgpr_read_b32 %11, a27\n\tv_accvgpr_read_b32 %12, a28\n\tv_accvgpr_read_b32 %13, a29\n\tv_accvgpr_read_b32 %14, a30\n\tv_accvgpr_read_b32 %15, a31" : "=v"(d[0]), "=v"(d[1]), "=v"(d[2]), "=v"(d[3]), "=v"(d[4]), "=v"(d[5]), "=v"(d[6]), "=v"(d[7]), "=v"(d[8]), "=v"(d[9]), "=v"(d[10]), "=v"(d[11]), "=v"(d[12]), "=v"(d[13]), "=v"(d[14]), "=v"(d[15])); PACK_STORE(2) }
        { float d[16]; asm volatile("v_accvgpr_read_b32 %0, a32\n\tv_accvgpr_read_b32 %1, a33\n\tv_accvgpr_read_b32 %2, a34\n\tv_accvgpr_read_b32 %3, a35\n\tv_accvgpr_read_b32 %4, a36\n\tv_accvgpr_read_b32 %5, a37\n\tv_accvgpr_read_b32 %6, a38\n\tv_accvgpr_read_b32 %7, a39\n\tv_accvgpr_read_b32 %8, a40\n\tv_accvgpr_read_b32 %9, a41\n\tv_accvgpr_read_b32 %10, a42\n\tv_accvgpr_read_b32 %11, a43\n\tv_accvgpr_read_b32 %12, a44\n\tv_accvgpr_read_b32 %13, a45\n\tv_accvgpr_read_b32 %14, a46\n\tv_accvgpr_read_b32 %15, a47" : "=v"(d[0]), "=v"(d[1]), "=v"(d[2]), "=v"(d[3]), "=v"(d[4]), "=v"(d[5]), "=v"(d[6]), "=v"(d[7]), "=v"(d[8]), "=v"(d[9]), "=v"(d[10]), "=v"(d[11]), "=v"(d[12]), "=v"(d[13]), "=v"(d[14]), "=v"(d[15])); PACK_STORE(4) }
        { float d[16]; asm volatile("v_accvgpr_read_b32 %0, a48\n\tv_accvgpr_read_b32 %1, a49\n\tv_accvgpr_read_b32 %2, a50\n\tv_accvgpr_read_b32 %3, a51\n\tv_accvgpr_read_b32 %4, a52\n\tv_accvgpr_read_b32 %5, a53\n\tv_accvgpr_read_b32 %6, a54\n\tv_accvgpr_read_b32 %7, a55\n\tv_accvgpr_read_b32 %8, a56\n\tv_accvgpr_read_b32 %9, a57\n\tv_accvgpr_read_b32 %10, a58\n\tv_accvgpr_read_b32 %11, a59\n\tv_accvgpr_read_b32 %12, a60\n\tv_accvgpr_read_b32 %13, a61\n\tv_accvgpr_read_b32 %14, a62\n\tv_accvgpr_read_b32 %15, a63" : "=v"(d[0]), "=v"(d[1]), "=v"(d[2]), "=v"(d[3]), "=v"(d[4]), "=v"(d[5]), "=v"(d[6]), "=v"(d[7]), "=v"(d[8]), "=v"(d[9]), "=v"(d[10]), "=v"(d[11]), "=v"(d[12]), "=v"(d[13]), "=v"(d[14]), "=v"(d[15])); PACK_STORE(6) }
        asm volatile("v_accvgpr_write_b32 a0, 0\n\tv_accvgpr_write_b32 a1, 0\n\tv_accvgpr_write_b32 a2, 0\n\tv_accvgpr_write_b32 a3, 0\n\tv_accvgpr_write_b32 a4, 0\n\tv_accvgpr_write_b32 a5, 0\n\tv_accvgpr_write_b32 a6, 0\n\tv_accvgpr_write_b32 a7, 0\n\tv_accvgpr_write_b32 a8, 0\n\tv_accvgpr_write_b32 a9, 0\n\tv_accvgpr_write_b32 a10, 0\n\tv_accvgpr_write_b32 a11, 0\n\tv_accvgpr_write_b32 a12, 0\n\tv_accvgpr_write_b32 a13, 0\n\tv_accvgpr_write_b32 a14, 0\n\tv_accvgpr_write_b32 a15, 0\n\tv_accvgpr_write_b32 a16, 0\n\tv_accvgpr_write_b32 a17, 0\n\tv_accvgpr_write_b32 a18, 0\n\tv_accvgpr_write_b32 a19, 0\n\tv_accvgpr_write_b32 a20, 0\n\tv_accvgpr_write_b32 a21, 0\n\tv_accvgpr_write_b32 a22, 0\n\tv_accvgpr_write_b32 a23, 0\n\tv_accvgpr_write_b32 a24, 0\n\tv_accvgpr_write_b32 a25, 0\n\tv_accvgpr_write_b32 a26, 0\n\tv_accvgpr_write_b32 a27, 0\n\tv_accvgpr_write_b32 a28, 0\n\tv_accvgpr_write_b32 a29, 0\n\tv_accvgpr_write_b32 a30, 0\n\tv_accvgpr_write_b32 a31, 0\n\tv_accvgpr_write_b32 a32, 0\n\tv_accvgpr_write_b32 a33, 0\n\tv_accvgpr_write_b32 a34, 0\n\tv_accvgpr_write_b32 a35, 0\n\tv_accvgpr_write_b32 a36, 0\n\tv_accvgpr_write_b32 a37, 0\n\tv_accvgpr_write_b32 a38, 0\n\tv_accvgpr_write_b32 a39, 0\n\tv_accvgpr_write_b32 a40, 0\n\tv_accvgpr_write_b32 a41, 0\n\tv_accvgpr_write_b32 a42, 0\n\tv_accvgpr_write_b32 a43, 0\n\tv_accvgpr_write_b32 a44, 0\n\tv_accvgpr_write_b32 a45, 0\n\tv_accvgpr_write_b32 a46, 0\n\tv_accvgpr_write_b32 a47, 0\n\tv_accvgpr_write_b32 a48, 0\n\tv_accvgpr_write_b32 a49, 0\n\tv_accvgpr_write_b32 a50, 0\n\tv_accvgpr_write_b32 a51, 0\n\tv_accvgpr_write_b32 a52, 0\n\tv_accvgpr_write_b32 a53, 0\n\tv_accvgpr_write_b32 a54, 0\n\tv_accvgpr_write_b32 a55, 0\n\tv_accvgpr_write_b32 a56, 0\n\tv_accvgpr_write_b32 a57, 0\n\tv_accvgpr_write_b32 a58, 0\n\tv_accvgpr_write_b32 a59, 0\n\tv_accvgpr_write_b32 a60, 0\n\tv_accvgpr_write_b32 a61, 0\n\tv_accvgpr_write_b32 a62, 0\n\tv_accvgpr_write_b32 a63, 0" ::: "a0", "a1", "a2", "a3", "a4", "a5", "a6", "a7", "a8", "a9", "a10", "a11", "a12", "a13", "a14", "a15", "a16", "a17", "a18", "a19", "a20", "a21", "a22", "a23", "a24", "a25", "a26", "a27", "a28", "a29", "a30", "a31", "a32", "a33", "a34", "a35", "a36", "a37", "a38", "a39", "a40", "a41", "a42", "a43", "a44", "a45", "a46", "a47", "a48", "a49", "a50", "a51", "a52", "a53", "a54", "a55", "a56", "a57", "a58", "a59", "a60", "a61", "a62", "a63");
        asm volatile("s_waitcnt vmcnt(0)" ::: "memory");
        if (lane == 0) __builtin_amdgcn_raw_buffer_store_b32(seq, rp, 0, pflag, AUX_SYS);
        inA = 0; stop = total;
        if (wid == 0) STAMP(2);
      }
      if (ctl & want & 2) {
        const u32 so = OFF_P + tileB * 32768 + wid * PIECE;
        const u32 xe = PDIRECT ? lane * 16 : (lane >> 3) * 128 + (((lane & 7) ^ (lane >> 4)) << 4);
        asm volatile("buffer_load_dwordx4 a[64:67], %0, %2, %3 offen offset:0 sc0 sc1\n\tbuffer_load_dwordx4 a[68:71], %1, %2, %3 offen offset:1024 sc0 sc1\n\tbuffer_load_dwordx4 a[72:75], %0, %2, %3 offen offset:2048 sc0 sc1\n\tbuffer_load_dwordx4 a[76:79], %1, %2, %3 offen offset:3072 sc0 sc1\n\tbuffer_load_dwordx4 a[80:83], %0, %2, %4 offen offset:0 sc0 sc1\n\tbuffer_load_dwordx4 a[84:87], %1, %2, %4 offen offset:1024 sc0 sc1\n\tbuffer_load_dwordx4 a[88:91], %0, %2, %4 offen offset:2048 sc0 sc1\n\tbuffer_load_dwordx4 a[92:95], %1, %2, %4 offen offset:3072 sc0 sc1" :: "v"(xe), "v"(PDIRECT ? xe : xe ^ 64), "s"(rs), "s"(so), "s"(so + 4096) : "memory", "a64", "a65", "a66", "a67", "a68", "a69", "a70", "a71", "a72", "a73", "a74", "a75", "a76", "a77", "a78", "a79", "a80", "a81", "a82", "a83", "a84", "a85", "a86", "a87", "a88", "a89", "a90", "a91", "a92", "a93", "a94", "a95");
        want = 1;
      }
      if (done == total) {
        if (want == 1) break;
        stop = total + 1;
      }
    }
    if (hasB) {
      asm volatile("s_waitcnt vmcnt(0)" ::: "memory");
      { float* d = y; asm volatile("v_accvgpr_read_b32 %0, a0\n\tv_accvgpr_read_b32 %1, a1\n\tv_accvgpr_read_b32 %2, a2\n\tv_accvgpr_read_b32 %3, a3\n\tv_accvgpr_read_b32 %4, a4\n\tv_accvgpr_read_b32 %5, a5\n\tv_accvgpr_read_b32 %6, a6\n\tv_accvgpr_read_b32 %7, a7\n\tv_accvgpr_read_b32 %8, a8\n\tv_accvgpr_read_b32 %9, a9\n\tv_accvgpr_read_b32 %10, a10\n\tv_accvgpr_read_b32 %11, a11\n\tv_accvgpr_read_b32 %12, a12\n\tv_accvgpr_read_b32 %13, a13\n\tv_accvgpr_read_b32 %14, a14\n\tv_accvgpr_read_b32 %15, a15" : "=v"(d[0]), "=v"(d[1]), "=v"(d[2]), "=v"(d[3]), "=v"(d[4]), "=v"(d[5]), "=v"(d[6]), "=v"(d[7]), "=v"(d[8]), "=v"(d[9]), "=v"(d[10]), "=v"(d[11]), "=v"(d[12]), "=v"(d[13]), "=v"(d[14]), "=v"(d[15])); }
      { float* d = y + 16; asm volatile("v_accvgpr_read_b32 %0, a16\n\tv_accvgpr_read_b32 %1, a17\n\tv_accvgpr_read_b32 %2, a18\n\tv_accvgpr_read_b32 %3, a19\n\tv_accvgpr_read_b32 %4, a20\n\tv_accvgpr_read_b32 %5, a21\n\tv_accvgpr_read_b32 %6, a22\n\tv_accvgpr_read_b32 %7, a23\n\tv_accvgpr_read_b32 %8, a24\n\tv_accvgpr_read_b32 %9, a25\n\tv_accvgpr_read_b32 %10, a26\n\tv_accvgpr_read_b32 %11, a27\n\tv_accvgpr_read_b32 %12, a28\n\tv_accvgpr_read_b32 %13, a29\n\tv_accvgpr_read_b32 %14, a30\n\tv_accvgpr_read_b32 %15, a31" : "=v"(d[0]), "=v"(d[1]), "=v"(d[2]), "=v"(d[3]), "=v"(d[4]), "=v"(d[5]), "=v"(d[6]), "=v"(d[7]), "=v"(d[8]), "=v"(d[9]), "=v"(d[10]), "=v"(d[11]), "=v"(d[12]), "=v"(d[13]), "=v"(d[14]), "=v"(d[15])); }
      { float* d = y + 32; asm volatile("v_accvgpr_read_b32 %0, a32\n\tv_accvgpr_read_b32 %1, a33\n\tv_accvgpr_read_b32 %2, a34\n\tv_accvgpr_read_b32 %3, a35\n\tv_accvgpr_read_b32 %4, a36\n\tv_accvgpr_read_b32 %5, a37\n\tv_accvgpr_read_b32 %6, a38\n\tv_accvgpr_read_b32 %7, a39\n\tv_accvgpr_read_b32 %8, a40\n\tv_accvgpr_read_b32 %9, a41\n\tv_accvgpr_read_b32 %10, a42\n\tv_accvgpr_read_b32 %11, a43\n\tv_accvgpr_read_b32 %12, a44\n\tv_accvgpr_read_b32 %13, a45\n\tv_accvgpr_read_b32 %14, a46\n\tv_accvgpr_read_b32 %15, a47" : "=v"(d[0]), "=v"(d[1]), "=v"(d[2]), "=v"(d[3]), "=v"(d[4]), "=v"(d[5]), "=v"(d[6]), "=v"(d[7]), "=v"(d[8]), "=v"(d[9]), "=v"(d[10]), "=v"(d[11]), "=v"(d[12]), "=v"(d[13]), "=v"(d[14]), "=v"(d[15])); }
      { float* d = y + 48; asm volatile("v_accvgpr_read_b32 %0, a48\n\tv_accvgpr_read_b32 %1, a49\n\tv_accvgpr_read_b32 %2, a50\n\tv_accvgpr_read_b32 %3, a51\n\tv_accvgpr_read_b32 %4, a52\n\tv_accvgpr_read_b32 %5, a53\n\tv_accvgpr_read_b32 %6, a54\n\tv_accvgpr_read_b32 %7, a55\n\tv_accvgpr_read_b32 %8, a56\n\tv_accvgpr_read_b32 %9, a57\n\tv_accvgpr_read_b32 %10, a58\n\tv_accvgpr_read_b32 %11, a59\n\tv_accvgpr_read_b32 %12, a60\n\tv_accvgpr_read_b32 %13, a61\n\tv_accvgpr_read_b32 %14, a62\n\tv_accvgpr_read_b32 %15, a63" : "=v"(d[0]), "=v"(d[1]), "=v"(d[2]), "=v"(d[3]), "=v"(d[4]), "=v"(d[5]), "=v"(d[6]), "=v"(d[7]), "=v"(d[8]), "=v"(d[9]), "=v"(d[10]), "=v"(d[11]), "=v"(d[12]), "=v"(d[13]), "=v"(d[14]), "=v"(d[15])); }
      u32 mx = 0;
#define ADD_P(O)                                                                                    \
      _Pragma("unroll") for (int i = 0; i < 16; ++i) {                                              \
        y[O + 2 * i] += __uint_as_float(d[i] << 16);                                                \
        y[O + 2 * i + 1] += __uint_as_float(d[i] & 0xffff0000u);                                    \
        mx = max(mx, max(__float_as_uint(y[O + 2 * i]) & 0x7fffffffu, __float_as_uint(y[O + 2 * i + 1]) & 0x7fffffffu)); \
      }
      { u32 d[16]; asm volatile("v_accvgpr_read_b32 %0, a64\n\tv_accvgpr_read_b32 %1, a65\n\tv_accvgpr_read_b32 %2, a66\n\tv_accvgpr_read_b32 %3, a67\n\tv_accvgpr_read_b32 %4, a68\n\tv_accvgpr_read_b32 %5, a69\n\tv_accvgpr_read_b32 %6, a70\n\tv_accvgpr_read_b32 %7, a71\n\tv_accvgpr_read_b32 %8, a72\n\tv_accvgpr_read_b32 %9, a73\n\tv_accvgpr_read_b32 %10, a74\n\tv_accvgpr_read_b32 %11, a75\n\tv_accvgpr_read_b32 %12, a76\n\tv_accvgpr_read_b32 %13, a77\n\tv_accvgpr_read_b32 %14, a78\n\tv_accvgpr_read_b32 %15, a79" : "=v"(d[0]), "=v"(d[1]), "=v"(d[2]), "=v"(d[3]), "=v"(d[4]), "=v"(d[5]), "=v"(d[6]), "=v"(d[7]), "=v"(d[8]), "=v"(d[9]), "=v"(d[10]), "=v"(d[11]), "=v"(d[12]), "=v"(d[13]), "=v"(d[14]), "=v"(d[15])); ADD_P(0) }
      { u32 d[16]; asm volatile("v_accvgpr_read_b32 %0, a80\n\tv_accvgpr_read_b32 %1, a81\n\tv_accvgpr_read_b32 %2, a82\n\tv_accvgpr_read_b32 %3, a83\n\tv_accvgpr_read_b32 %4, a84\n\tv_accvgpr_read_b32 %5, a85\n\tv_accvgpr_read_b32 %6, a86\n\tv_accvgpr_read_b32 %7, a87\n\tv_accvgpr_read_b32 %8, a88\n\tv_accvgpr_read_b32 %9, a89\n\tv_accvgpr_read_b32 %10, a90\n\tv_accvgpr_read_b32 %11, a91\n\tv_accvgpr_read_b32 %12, a92\n\tv_accvgpr_read_b32 %13, a93\n\tv_accvgpr_read_b32 %14, a94\n\tv_accvgpr_read_b32 %15, a95" : "=v"(d[0]), "=v"(d[1]), "=v"(d[2]), "=v"(d[3]), "=v"(d[4]), "=v"(d[5]), "=v"(d[6]), "=v"(d[7]), "=v"(d[8]), "=v"(d[9]), "=v"(d[10]), "=v"(d[11]), "=v"(d[12]), "=v"(d[13]), "=v"(d[14]), "=v"(d[15])); ADD_P(32) }
#pragma unroll
      for (int s = 32; s >= 1; s >>= 1) mx = max(mx, (u32)__shfl_xor((int)mx, s));
      if (lane == 0) lds_st32(C_WMAX + wid * 4, mx);
      if (wid == 0) STAMP(3);
    }
   }
  } else {
    const u32 lw = wid - 4;
    if (ABL != 4) send_stream<false>(p, rank, (b - 8 * NSJ) + (256 - 8 * NSJ) * lw, 4 * (256 - 8 * NSJ), seqv, lane, t0);
    if (lw == 0) STAMP(7);
    const u32 seq = rfl(seqv) + 1;
    u32 n = 0;
    bool ok = true;
    if (ABL != 1) {
      if (hasA) ok = load_job(lw, 0, UA, vA, n, myws, lane, seq, t0);
      if (hasA && hasB) {
        for (u32 fin = !ok; !fin;) {
          u32 c;
          asm volatile("ds_read_b32 %0, %1\n\ts_waitcnt lgkmcnt(0)" : "=v"(c) : "v"(C_CONS + (lane & 15) * 4) : "memory");
          fin = __all(c >= 8);
          if (__builtin_amdgcn_s_memrealtime() - t0 > DEADLINE) { ok = false; fin = 1; }
        }
      }
      if (hasB && ok) ok = load_job(lw, 1, UB, vB, n, myws, lane, seq, t0);
      if (!ok) asm volatile("ds_or_b32 %0, %1" ::"v"(C_CTL), "v"(1u) : "memory");
    }
  }
  asm volatile("s_waitcnt lgkmcnt(0)" ::: "memory");
  __syncthreads();
  const u32 seq = rfl(seqv) + 1;
  if (threadIdx.x == 0) __hip_atomic_store((u32*)(myws + OFF_CNT + b * 64), seq, __ATOMIC_RELAXED, __HIP_MEMORY_SCOPE_AGENT);
  if (!hasB) return;
  if (ABL == 4) {
    if (wid < 4 && y[0] == 123.456f) p.out[0][0] = 1;
    return;
  }
  {
    const u32x4 wmx = *(const u32x4*)(lds + C_WMAX);
    const u32 m = max(max(wmx[0], wmx[1]), max(wmx[2], wmx[3]));
    if (lane == 0)
      __hip_atomic_store((u64*)(p.ws[wid] + OFF_G) + rank * 128 + tileB, ((u64)seq << 32) | m, __ATOMIC_RELAXED, __HIP_MEMORY_SCOPE_SYSTEM);
    const rsrc_t rs = mk_rsrc(myws, WS_USED);
    u32 gm;
    for (;;) {
      const u32x4 g0 = __builtin_amdgcn_raw_buffer_load_b128(rs, lane * 16, OFF_G + wid * 1024, AUX_SYS);
      gm = max(g0[0], g0[2]);
      if (__all(g0[1] == seq && g0[3] == seq)) break;
      if (__builtin_amdgcn_s_memrealtime() - t0 > DEADLINE) { lds_st32(C_CTL, 1); break; }
      asm volatile("" ::: "memory");
    }
#pragma unroll
    for (int s = 32; s >= 1; s >>= 1) gm = max(gm, (u32)__shfl_xor((int)gm, s));
    if (lane == 0) lds_st32(C_GMAX + wid * 4, gm);
  }
  asm volatile("s_waitcnt lgkmcnt(0)" ::: "memory");
  __syncthreads();
  if (wid == 0) STAMP(4);
  if (wid < 4) {
    const u32x4 ga = *(const u32x4*)(lds + C_GMAX), gb = *(const u32x4*)(lds + C_GMAX + 16);
    const u32 ab0 = *(const u32*)(lds + C_CTL) & 1;
    const float amax = __uint_as_float(max(max(max(ga[0], ga[1]), max(ga[2], ga[3])), max(max(gb[0], gb[1]), max(gb[2], gb[3]))));
    const float scale = amax / 448.0f;
    const bool ab = ab0 || amax != amax;
    const float rinv = scale > 0.f || scale != scale ? 1.0f / scale : 0.f;
    const u32 h = lane >> 5;
#pragma unroll
    for (int t = 0; t < 4; ++t) {
      u32x4 o4;
#pragma unroll
      for (int e = 0; e < 4; ++e) {
        float f[4];
#pragma unroll
        for (int k = 0; k < 4; ++k) {
          const float yy = y[16 * t + 4 * e + k], z0 = yy * rinv;
          f[k] = __builtin_amdgcn_fmed3f(__builtin_fmaf(__builtin_fmaf(-z0, scale, yy), rinv, z0), -448.f, 448.f);
        }
        int pk = __builtin_amdgcn_cvt_pk_fp8_f32(f[0], f[1], 0, false);
        pk = __builtin_amdgcn_cvt_pk_fp8_f32(f[2], f[3], pk, true);
        o4[e] = ab ? 0x7f7f7f7fu : (u32)pk;
      }
      const u32 m = wm * 64 + (t >> 1) * 32 + (lane & 31), ch = 4 * wn + 2 * (t & 1) + h;
      *(u32x4*)(lds + m * 128 + ((ch ^ (m & 7)) << 4)) = o4;
    }
  }
  __syncthreads();
  if (wid == 0) STAMP(1);
  {
    const u32 pq = (wid + b + (b >> 3)) & 7;
    const u32 so = row0_of(rank, 2 * UB, 1) * 4096 + col0_of(rank, vB);
    const u32 r8 = lane >> 3, c = lane & 7;
    const rsrc_t ro = mk_rsrc(p.out[pq], 16 * MiB);
#pragma unroll
    for (int t = 0; t < 16; ++t) {
      const u32 row = 8 * t + r8;
      const u32x4 o4 = *(const u32x4*)(lds + row * 128 + ((c ^ (row & 7)) << 4));
      st128(o4, ro, row * 4096 + c * 16, so);
    }
    asm volatile("s_waitcnt vmcnt(0)" ::: "memory");
    if (wid == 0) STAMP(5);
    if (pq != rank) {
      if (lane == 0)
        __hip_atomic_store((u32*)(p.ws[pq] + OFF_FD + tileB * 64) + rank, seq, __ATOMIC_RELAXED, __HIP_MEMORY_SCOPE_SYSTEM);
      const u32* fd = (const u32*)(myws + OFF_FD + tileB * 64) + pq;
      while (__hip_atomic_load(fd, __ATOMIC_RELAXED, __HIP_MEMORY_SCOPE_SYSTEM) < seq) {
        if (__builtin_amdgcn_s_memrealtime() - t0 > DEADLINE) {
          if (lane == 0) p.out[rank][so] = 0x7f;
          break;
        }
        __builtin_amdgcn_s_sleep(8);
      }
    }
    if (pq == ((rank + 1) & 7)) STAMP(6);
  }
}

extern "C" void kernel_launch(int rank, int nranks, ncclComm_t comm, hipStream_t stream, void* const* d_in,
                              const int* in_sizes, int n_in, void* d_out, int out_size, const int* in_dtypes,
                              int out_dtype, void* const* d_out_all, void* const* d_ws_all, size_t ws_bytes) {
  if (nranks != 8 || ws_bytes < WS_USED || (unsigned)rank >= 8u) {
    fprintf(stderr, "unsupported configuration\n");
    return;
  }
  (void)hipFuncSetAttribute((const void*)gemm_ar_q8_fused, hipFuncAttributeMaxDynamicSharedMemorySize, LDS_BYTES);
  Params p;
  p.x = d_in[0];
  p.w = d_in[1];
  for (int i = 0; i < 8; ++i) {
    p.ws[i] = (char*)d_ws_all[i];
    p.out[i] = (unsigned char*)d_out_all[i];
  }
  p.rank = rank;
  gemm_ar_q8_fused<<<256, 512, LDS_BYTES, stream>>>(p);
}
```

```cpp
#include <hip/hip_runtime.h>
#include <rccl/rccl.h>
#include <cstdio>
#include <cstdint>

typedef unsigned int u32;
typedef unsigned long long u64;
typedef u32 u32x4 __attribute__((ext_vector_type(4)));
typedef u32 u32x2 __attribute__((ext_vector_type(2)));
typedef __bf16 bf16x2 __attribute__((ext_vector_type(2)));
typedef float f32x2 __attribute__((ext_vector_type(2)));
typedef __amdgpu_buffer_rsrc_t rsrc_t;

#define AFF 1
#define NSJ 4
#define SW 2
#define WR (AFF ? (NSJ * 8 / 7) * SW : NSJ * 8 * SW / 7)
#define GAUX 17
#define PSCOPE __HIP_MEMORY_SCOPE_SYSTEM
#define PRET 0
#define GSLEEP 0
#define PF 1
#define CPOLL 1
#define PSLEEP 1
#define SW3 SW
#define WR3 ((NSJ * 8 / 7) * SW3)
#define DIRN 2
#define VD 8
#define ABL 0

constexpr u32 MiB = 1u << 20;
constexpr u32 PIECE = 8192;
constexpr u32 OFF_XS = 0;
constexpr u32 OFF_XO = 4 * MiB;
constexpr u32 OFF_W = 8 * MiB;
constexpr u32 OFF_P = 16 * MiB;
constexpr u32 OFF_PO = 20 * MiB;
constexpr u32 OFF_FX = 24 * MiB;
constexpr u32 OFF_FW = OFF_FX + 4096;
constexpr u32 OFF_FP = OFF_FW + 4096;
constexpr u32 OFF_G = OFF_FP + 8192;
constexpr u32 OFF_FD = OFF_G + 8192;
constexpr u32 OFF_FT = OFF_FD + 8192;
constexpr u32 OFF_FE = OFF_FT + 512;
constexpr u32 OFF_CNT = OFF_FE + 1024 + 512;
constexpr u32 OFF_Q = OFF_CNT + 16384;
constexpr u32 OFF_DBG = OFF_Q + 1024;
constexpr u32 WS_USED = OFF_DBG + 256 * 128 * 2;

constexpr u32 SLOT = 4 * PIECE;
constexpr u32 CTRL = 4 * SLOT;
constexpr u32 LDS_BYTES = CTRL + 256;
constexpr u32 C_FULL = CTRL + 0;
constexpr u32 C_CTL = CTRL + 16;
constexpr u32 C_CONS = CTRL + 32;
constexpr u32 C_WMAX = CTRL + 96;
constexpr u32 C_GMAX = CTRL + 128;
static_assert(C_FULL == 0x20000 && SLOT == 0x8000, "literals in the asm loops");
constexpr u64 DEADLINE = 50000000ull;
constexpr int AUX_SYS = 17;

struct Params {
  const void* x;
  const void* w;
  char* ws[8];
  unsigned char* out[8];
  int rank;
};

__device__ __forceinline__ u32 rfl(u32 v) { return __builtin_amdgcn_readfirstlane(v); }
__device__ __forceinline__ rsrc_t mk_rsrc(const void* p, u32 bytes) {
  return __builtin_amdgcn_make_buffer_rsrc((void*)p, (short)0, (int)bytes, 0x00020000);
}
__device__ __forceinline__ void lds_st32(u32 addr, u32 v) { asm volatile("ds_write_b32 %0, %1" ::"v"(addr), "v"(v) : "memory"); }
__device__ __forceinline__ void st128(u32x4 v, rsrc_t r, u32 voff, u32 soff) {
  asm volatile("buffer_store_dwordx4 %0, %1, %2, %3 offen sc0 sc1\n\ts_nop 1" ::"v"(v), "v"(voff), "s"(r), "s"(soff) : "memory");
}
template <int OFF> __device__ __forceinline__ void st128i(u32x4 v, rsrc_t r, u32 voff, u32 soff) {
  asm volatile("buffer_store_dwordx4 %0, %1, %2, %3 offen offset:%4 sc0 sc1\n\ts_nop 1" ::"v"(v), "v"(voff), "s"(r), "s"(soff), "i"(OFF) : "memory");
}
__device__ __forceinline__ u32 wave_max(u32 x) {
  x = max(x, (u32)__builtin_amdgcn_update_dpp(0, (int)x, 0xB1, 0xf, 0xf, true));
  x = max(x, (u32)__builtin_amdgcn_update_dpp(0, (int)x, 0x4E, 0xf, 0xf, true));
  x = max(x, (u32)__builtin_amdgcn_update_dpp(0, (int)x, 0x141, 0xf, 0xf, true));
  x = max(x, (u32)__builtin_amdgcn_update_dpp(0, (int)x, 0x140, 0xf, 0xf, true));
  return max(max((u32)__builtin_amdgcn_readlane((int)x, 0), (u32)__builtin_amdgcn_readlane((int)x, 16)),
             max((u32)__builtin_amdgcn_readlane((int)x, 32), (u32)__builtin_amdgcn_readlane((int)x, 48)));
}
__device__ __forceinline__ void sgpr_pad(u32 s) { asm volatile("s_nop 4" ::"s"(s)); }
#define STAMP(i) do { } while (0)

__device__ __forceinline__ u32 row0_of(u32 r, u32 u, u32 own) {
  const u32 l = r >> 2, i = (r >> 1) & 1;
  const u32 quad = (l + 2 * i + (u >> 3)) & 3;
  const u32 half = own ? l : 1 - l;
  return quad * 1024 + half * 512 + (u & 7) * 64;
}
__device__ __forceinline__ u32 col0_of(u32 r, u32 v) {
  const u32 l = r >> 2, j = r & 1;
  return ((l + 2 * j + (v >> 3)) & 3) * 1024 + (v & 7) * 128;
}

__constant__ const u32 w_order[80] = {0, 8, 1, 16, 9, 2, 24, 17, 10, 32, 3, 25, 18, 40, 11, 33, 4, 26, 48, 19, 41, 12, 34, 56, 27, 49, 20, 42, 64, 35, 57, 28, 50, 72, 43, 65, 36, 58, 80, 51, 73, 44, 66, 88, 59, 81, 52, 74, 96, 67, 89, 60, 82, 104, 75, 97, 68, 90, 112, 83, 105, 76, 98, 120, 91, 113, 84, 106, 99, 121, 92, 114, 107, 100, 122, 115, 108, 123, 116, 124};
struct PieceD { u32 kind, src, dst, flag, dep; };
__device__ __forceinline__ PieceD decode(u32 q, u32 rank, u32 dest, u32 nk, u32 kbase, u32 step0) {
  PieceD d;
  const u32 nx = 16 * nk;
  d.dep = OFF_FP;
  if (q >= 4 * nx) {
    const u32 qq = q - 4 * nx, l = rank >> 2, l2 = 1 - l;
    const u32 qa0 = (l + (rank & 2) - l2) & 3, qb0 = (l + 2 * (rank & 1) - l2) & 3;
    const u32 uh = (qa0 >> 1) != ((dest >> 1) & 1), vh = (qb0 >> 1) != (dest & 1);
    const u32 qa = (qa0 + uh) & 3, qb = (qb0 + vh) & 3;
    const u32 ul = (qq >> 2) & 3, vl = qq >> 4, h = qq & 3;
    const u32 tile = (uh * 4 + ul) * 16 + vh * 8 + vl, ptile = ((qa & 1) * 4 + ul) * 16 + (qb & 1) * 8 + vl;
    d.kind = 2;
    d.src = OFF_PO + tile * 32768 + h * PIECE;
    d.dst = OFF_P + ptile * 32768 + h * PIECE;
    d.flag = OFF_FP + ptile * 64 + h * 4;
    d.dep = OFF_FP + tile * 64 + 32 + h * 4;
  } else if (q >= nx && q < 3 * nx) {
    const u32 qq = q - nx;
    const u32 e5 = w_order[rfl(min(qq >> 1, 79u))];
    const u32 o = (nk == 5) ? e5 >> 3 : qq / 6, ks = (nk == 5) ? e5 & 7 : (qq - o * 6) >> 1;
    const u32 half = qq & 1, v = (o >> 1) + 8 * (o & 1), step = step0 + ks;
    d.kind = 1;
    d.src = (kbase + 64 * ks) * 8192 + (col0_of(dest, v) + 64 * half) * 2;
    d.dst = OFF_W + ((v * 32 + step) * 2 + half) * PIECE;
    d.flag = OFF_FW + ((2 * v + half) * 32 + step) * 4;
  } else {
    const u32 own = q >= nx, qq = own ? q - 3 * nx : q;
    const u32 ks = qq >> 4, u = qq & 15, step = step0 + ks;
    d.kind = 0;
    d.src = row0_of(dest, u, own) * 1024 + (kbase + 64 * ks) * 2;
    d.dst = (own ? OFF_XO : OFF_XS) + (u * 32 + step) * PIECE;
    d.flag = OFF_FX + ((own * 16 + u) * 32 + step) * 4;
  }
  d.kind = rfl(d.kind); d.src = rfl(d.src); d.dst = rfl(d.dst); d.flag = rfl(d.flag); d.dep = rfl(d.dep);
  return d;
}

__device__ __forceinline__ void piece_load(u32x4 (&v)[8], const PieceD& d, rsrc_t rx, rsrc_t rw, rsrc_t rl, u32 lane) {
  const u32 kind = rfl(d.kind);
  const u32 vo = (lane >> 3) * (kind == 1 ? 65536u : kind == 0 ? 1024u : 128u) + (lane & 7) * 16;
  const rsrc_t r = kind == 1 ? rw : kind == 0 ? rx : rl;
  const u32 st = rfl(kind == 2 ? 1024u : 8192u);
  u32 so = rfl(d.src);
  asm volatile("s_nop 4\n\tbuffer_load_dwordx4 %0, %9, %10, %8 offen sc1\n\ts_add_u32 %8, %8, %11\n\t"
               "buffer_load_dwordx4 %1, %9, %10, %8 offen sc1\n\ts_add_u32 %8, %8, %11\n\t"
               "buffer_load_dwordx4 %2, %9, %10, %8 offen sc1\n\ts_add_u32 %8, %8, %11\n\t"
               "buffer_load_dwordx4 %3, %9, %10, %8 offen sc1\n\ts_add_u32 %8, %8, %11\n\t"
               "buffer_load_dwordx4 %4, %9, %10, %8 offen sc1\n\ts_add_u32 %8, %8, %11\n\t"
               "buffer_load_dwordx4 %5, %9, %10, %8 offen sc1\n\ts_add_u32 %8, %8, %11\n\t"
               "buffer_load_dwordx4 %6, %9, %10, %8 offen sc1\n\ts_add_u32 %8, %8, %11\n\t"
               "buffer_load_dwordx4 %7, %9, %10, %8 offen sc1"
               : "=&v"(v[0]), "=&v"(v[1]), "=&v"(v[2]), "=&v"(v[3]), "=&v"(v[4]), "=&v"(v[5]), "=&v"(v[6]), "=&v"(v[7]), "+&s"(so)
               : "v"(vo), "s"(r), "s"(st) : "scc", "memory");
}
__device__ __forceinline__ void dep_load(u32& f, u32 dep, rsrc_t rl) {
  asm volatile("s_nop 4\n\tbuffer_load_dword %0, off, %1, %2 sc0 sc1" : "=&v"(f) : "s"(rl), "s"(rfl(dep)) : "memory");
}
__device__ __forceinline__ void claim(u32& c, u32 qoff, rsrc_t rl) {
  c = 1;
  asm volatile("s_mov_b64 exec, 1\n\tbuffer_atomic_add %0, off, %1, %2 sc0\n\ts_mov_b64 exec, -1" : "+v"(c) : "s"(rl), "s"(qoff) : "memory");
}
#define WAITV(N, v, f, c) asm volatile("s_waitcnt vmcnt(" #N ")" : "+v"(v[0]), "+v"(v[1]), "+v"(v[2]), "+v"(v[3]), "+v"(v[4]), "+v"(v[5]), "+v"(v[6]), "+v"(v[7]), "+v"(f), "+v"(c) :: "memory")
template <int E> __device__ __forceinline__ void w_store(const u32x4 (&v)[8], rsrc_t rd, u32 wo, u32 dst) {
  u32x4 o;
#pragma unroll
  for (int j = 0; j < 4; ++j)
    o[j] = __builtin_amdgcn_perm(v[2 * j + 1][E >> 1], v[2 * j][E >> 1], (E & 1) ? 0x07060302u : 0x05040100u);
  st128i<((E & 3) + 8 * (E >> 2)) * 128>(o, rd, wo ^ ((((E >> 1) & 1) + 4 * (E >> 2)) << 4), dst);
}
__device__ __forceinline__ void piece_store(const u32x4 (&v)[8], const PieceD& d, rsrc_t rd, u32 lane) {
  const u32 hi = lane >> 3, lo = lane & 7, dst = rfl(d.dst), kind = rfl(d.kind);
  sgpr_pad(dst);
  if (kind == 1) {
    const u32 wo = (32 * (lo >> 2) + 16 * (lo & 1) + 4 * ((lo >> 1) & 1)) * 128 + ((hi ^ (lo & 2)) << 4);
    w_store<0>(v, rd, wo, dst); w_store<1>(v, rd, wo, dst); w_store<2>(v, rd, wo, dst); w_store<3>(v, rd, wo, dst);
    w_store<4>(v, rd, wo, dst); w_store<5>(v, rd, wo, dst); w_store<6>(v, rd, wo, dst); w_store<7>(v, rd, wo, dst);
  } else {
    const u32 xe = hi * 128 + ((lo ^ (hi >> 1)) << 4), xo = xe ^ 64, dst2 = dst + 4096;
    st128i<0>(v[0], rd, xe, dst); st128i<1024>(v[1], rd, xo, dst); st128i<2048>(v[2], rd, xe, dst); st128i<3072>(v[3], rd, xo, dst);
    st128i<0>(v[4], rd, xe, dst2); st128i<1024>(v[5], rd, xo, dst2); st128i<2048>(v[6], rd, xe, dst2); st128i<3072>(v[7], rd, xo, dst2);
  }
}

template <bool DYN>
__device__ __forceinline__ void send_stream(const Params& p, u32 dest, u32 q0, u32 qs, u32 cntoff, u32 lane, u64 t0) {
  const u32 rank = p.rank;
  const bool same = (dest >> 2) == (rank >> 2);
  const u32 nk = same ? 5 : 3, kbase = same ? 0 : 320, step0 = same ? (rank & 3) * 5 : 20 + (rank & 3) * 3;
  const u32 total = (ABL == 1 || same) ? 64 * nk : 64 * nk + 16 * VD;
  if (q0 >= total) return;
  const rsrc_t rx = mk_rsrc(p.x, 4 * MiB), rw = mk_rsrc(p.w, 4 * MiB), rd = mk_rsrc(p.ws[dest], WS_USED), rl = mk_rsrc(p.ws[rank], WS_USED);
  u32x4 bufA[8], bufB[8];
  u32 f = 0;
  u32 cl = 0, i1 = q0 + qs, i2 = q0 + 2 * qs;
  PieceD dc = decode(q0, rank, dest, nk, kbase, step0);
  dep_load(f, cntoff, rl);
  piece_load(bufA, dc, rx, rw, rl, lane);
  PieceD dn = decode(min(i1, total - 1), rank, dest, nk, kbase, step0);
  piece_load(bufB, dn, rx, rw, rl, lane);
  WAITV(8, bufA, f, cl);
  const u32 seq = rfl(f) + 1;
  const u32 qoff = OFF_Q + ((seq & 1) * 8 + dest) * 64;
  u32 prevflag = 0;
  bool rdyn = true;
#define FLAG(off) __builtin_amdgcn_raw_buffer_store_b32(seq, rd, 0, rfl(off), AUX_SYS)
#define SEND_HALF(cur, nxt, LD)                                                               \
  {                                                                                           \
    const bool more = i1 < total;                                                             \
    const PieceD d2 = decode(min(i2, total - 1), rank, dest, nk, kbase, step0);               \
    if (PF && more && !rdyn) {                    \
      u32 g;                                                                                  \
      dep_load(g, dn.dep, rl);                                                                \
      asm volatile("s_waitcnt vmcnt(0)" : "+v"(g) :: "memory");                               \
      if (prevflag && lane == 0) FLAG(prevflag);                                              \
      prevflag = 0;                                                                           \
      rdyn = rfl(g) >= seq;                                                                   \
    }                                                                                         \
    const u32 slow = rfl(more && !rdyn);                                                      \
    if (slow) {                                                                               \
      piece_store(cur, dc, rd, lane);                                                         \
      asm volatile("s_waitcnt vmcnt(0)" ::: "memory");                                        \
      if (lane == 0) { if (prevflag) FLAG(prevflag); FLAG(dc.flag); }                         \
      prevflag = 0;                                                                           \
      for (;;) {                                                                              \
        u32 g;                                                                                \
        dep_load(g, dn.dep, rl);                                                              \
        asm volatile("s_waitcnt vmcnt(0)" : "+v"(g) :: "memory");                             \
        if (rfl(g) >= seq) break;                                                             \
        if (__builtin_amdgcn_s_memrealtime() - t0 > DEADLINE) return;     \
        __builtin_amdgcn_s_sleep(PSLEEP);                                                     \
      }                                                                                       \
    }                                                                                         \
    if (DYN) claim(cl, qoff, rl);                                                             \
    dep_load(f, d2.dep, rl);                                                                  \
    if (LD) piece_load(nxt, dn, rx, rw, rl, lane);         \
    if (!slow) piece_store(cur, dc, rd, lane);                                                \
    else asm volatile("s_waitcnt vmcnt(0)" ::: "memory");                                     \
    WAITV(8, nxt, f, cl);                                                                      \
    if (prevflag && lane == 0) FLAG(prevflag);                                                \
    prevflag = slow ? 0u : dc.flag;                                                           \
    if (!more) break;                                                                         \
    rdyn = d2.kind != 2 || rfl(f) >= seq;                                       \
    dc = dn; dn = d2;                                                                         \
    i1 = i2; i2 = DYN ? 3 * qs + rfl(cl) : i2 + qs;                                           \
  }
  for (;;) {
    SEND_HALF(bufA, bufB, false)
    for (;;) {
      SEND_HALF(bufB, bufA, true)
      SEND_HALF(bufA, bufB, true)
    }
    break;
  }
  asm volatile("s_waitcnt vmcnt(0)" ::: "memory");
  if (prevflag && lane == 0) __builtin_amdgcn_raw_buffer_store_b32(seq, rd, 0, rfl(prevflag), AUX_SYS);
}

__device__ __forceinline__ u32 out_off(u32 q, u32 rank, u32 rot) {
  constexpr u32 NB = 8 - DIRN, NP = 8 * 2 * NB * 2;
  u32 r = min(q, NP - 1) + rot;
  r = r >= NP ? r - NP : r;
  const u32 tl = r >> 1, U = tl / (2 * NB), vv = tl - U * (2 * NB), v = vv < NB ? vv : vv + DIRN;
  return rfl(row0_of(rank, 2 * U + (r & 1), 1) * 4096 + col0_of(rank, v));
}
#define OUT8(op, v, r, so)                                                                    \
  asm volatile("s_nop 4\n\t" op " %0, %9, %10, %8 offen sc0 sc1\n\ts_add_u32 %8, %8, 0x8000\n\t"           \
               op " %1, %9, %10, %8 offen sc0 sc1\n\ts_add_u32 %8, %8, 0x8000\n\t"                         \
               op " %2, %9, %10, %8 offen sc0 sc1\n\ts_add_u32 %8, %8, 0x8000\n\t"                         \
               op " %3, %9, %10, %8 offen sc0 sc1\n\ts_add_u32 %8, %8, 0x8000\n\t"                         \
               op " %4, %9, %10, %8 offen sc0 sc1\n\ts_add_u32 %8, %8, 0x8000\n\t"                         \
               op " %5, %9, %10, %8 offen sc0 sc1\n\ts_add_u32 %8, %8, 0x8000\n\t"                         \
               op " %6, %9, %10, %8 offen sc0 sc1\n\ts_add_u32 %8, %8, 0x8000\n\t"                         \
               op " %7, %9, %10, %8 offen sc0 sc1\n\ts_nop 1"                                               \
               : "+v"(v[0]), "+v"(v[1]), "+v"(v[2]), "+v"(v[3]), "+v"(v[4]), "+v"(v[5]), "+v"(v[6]), "+v"(v[7]), "+&s"(so) \
               : "v"(vo), "s"(r) : "scc", "memory")
__device__ __forceinline__ void send_out(const Params& p, u32 dest, u32 j, u32 seq, u32 lane, u64 t0) {
  constexpr u32 NP = 8 * 2 * (8 - DIRN) * 2;
  const u32 rank = p.rank;
  const rsrc_t ro = mk_rsrc(p.out[rank], 16 * MiB), rd = mk_rsrc(p.out[dest], 16 * MiB), rl = mk_rsrc(p.ws[rank], WS_USED);
  const u32 qoff = OFF_Q + ((seq & 1) * 8 + dest) * 64 + 4, rot = ((rank * 8 + dest) * 37) % NP;
  const u32 vo = (lane >> 3) * 4096 + (lane & 7) * 16;
  bool ok = true;
  if (PRET) {
    const u32 a = __builtin_amdgcn_raw_buffer_load_b32(rd, (lane & 3) * 65536, row0_of(rank, lane >> 2, 1) * 4096, AUX_SYS);
    asm volatile("" ::"v"(a));
  }
  for (;;) {
    const u32x2 f = __builtin_amdgcn_raw_buffer_load_b64(rl, lane * 8, OFF_FT, AUX_SYS);
    const bool skip = ((2 * lane) & 7) >= 8 - DIRN, skip1 = ((2 * lane + 1) & 7) >= 8 - DIRN;
    if (__all((skip || f[0] >= seq) && (skip1 || f[1] >= seq))) break;
    if (__builtin_amdgcn_s_memrealtime() - t0 > DEADLINE) {
      if (lane == 0) p.out[rank][out_off(j, rank, rot)] = 0x7f;
      return;
    }
    asm volatile("" ::: "memory");
  }
  u32x4 bufA[8] = {}, bufB[8] = {};
  u32 cl = 0, dummy = 0, i1 = j + WR3, oc = out_off(j, rank, rot);
  { u32 so = oc; OUT8("buffer_load_dwordx4", bufA, ro, so); }
  { u32 so = out_off(i1, rank, rot); OUT8("buffer_load_dwordx4", bufB, ro, so); }
  WAITV(8, bufA, dummy, cl);
#define OUT_HALF(cur, nxt, LD)                                                                \
  {                                                                                           \
    const u32 on = out_off(i1, rank, rot);                                                    \
    claim(cl, qoff, rl);                                                                      \
    if (LD) { u32 so = on; OUT8("buffer_load_dwordx4", nxt, ro, so); }                        \
    { u32 so = oc; OUT8("buffer_store_dwordx4", cur, rd, so); }                               \
    WAITV(8, nxt, dummy, cl);                                                                 \
    if (i1 >= NP) break;                                                                      \
    oc = on; i1 = 2 * WR3 + rfl(cl);                                                          \
  }
  for (;;) {
    OUT_HALF(bufA, bufB, false)
    for (;;) {
      OUT_HALF(bufB, bufA, true)
      OUT_HALF(bufA, bufB, true)
    }
    break;
  }
  asm volatile("s_waitcnt vmcnt(0)" ::: "memory");
  if (lane == 0)
    __hip_atomic_store((u32*)(p.ws[dest] + OFF_FE) + rank * 32 + j, seq, __ATOMIC_RELAXED, __HIP_MEMORY_SCOPE_SYSTEM);
  const u32* fe = (const u32*)(p.ws[rank] + OFF_FE) + dest * 32 + j;
  while (ok && __hip_atomic_load(fe, __ATOMIC_RELAXED, __HIP_MEMORY_SCOPE_SYSTEM) < seq)
    if (__builtin_amdgcn_s_memrealtime() - t0 > DEADLINE) ok = false;
  if (!ok && lane == 0) p.out[rank][oc] = 0x7f;
}

#define GL4(so) "buffer_load_dwordx4 %[voff], %[rs], " so " offen sc1 lds\n\t"                 \
                "buffer_load_dwordx4 %[voff], %[rs], " so " offen offset:1024 sc1 lds\n\t"     \
                "buffer_load_dwordx4 %[voff], %[rs], " so " offen offset:2048 sc1 lds\n\t"     \
                "buffer_load_dwordx4 %[voff], %[rs], " so " offen offset:3072 sc1 lds\n\t"
#define NEXT4(so) "s_add_u32 m0, m0, 0x1000\n\ts_add_u32 " so ", " so ", 0x1000\n\t"
__device__ __forceinline__ bool load_job(u32 lw, u32 isB, u32 U, u32 v, u32& n, char* myws, u32 lane, u32 seq, u64 t0) {
  const rsrc_t rs = mk_rsrc(myws, WS_USED);
  const u32* fx = (const u32*)(myws + OFF_FX) + (isB * 16 + 2 * U) * 32 + lane;
  const u32* fw = (const u32*)(myws + OFF_FW) + (2 * v) * 32 + lane;
  const u32* fp = (const u32*)(myws + OFF_FP + (U * 16 + v) * 64) + (lane & 3);
  const u32 xbase = (isB ? OFF_XO : OFF_XS) + 2 * U * 32 * PIECE, wbase = OFF_W + v * 64 * PIECE;
  u32 rem = 0x11111111u << lw, rdy = 0, needP = isB && lw == 0;
  while (rem | needP) {
    const u32 cand = rdy & rem;
    if (!cand) {
      const u32 a = __hip_atomic_load(fx, __ATOMIC_RELAXED, PSCOPE);
      const u32 c = __hip_atomic_load(fw, __ATOMIC_RELAXED, PSCOPE);
      if (needP && __all(ABL == 4 || __hip_atomic_load(fp, __ATOMIC_RELAXED, PSCOPE) >= seq)) {
        asm volatile("ds_or_b32 %0, %1" ::"v"(C_CTL), "v"(lane == 0 ? 2u : 0u) : "memory");
        needP = 0;
      }
      const u64 m = __ballot(ABL == 4 || (a >= seq && c >= seq));
      rdy = (u32)m & (u32)(m >> 32);
      if (__builtin_amdgcn_s_memrealtime() - t0 > DEADLINE) return false;
      continue;
    }
    const u32 step = __builtin_ctz(cand);
    rem &= ~(1u << step);
    u32 x = xbase + step * PIECE, w = wbase + step * 2 * PIECE, tmp, spin = 0;
    ++n;
    asm volatile(
        "1:\n\t"
        "ds_read_b128 v[52:55], %[acons]\n\t"
        "s_waitcnt lgkmcnt(0)\n\t"
        "v_min_u32 v52, v52, v53\n\t"
        "v_min3_u32 v52, v52, v54, v55\n\t"
        "s_add_u32 %[spin], %[spin], 1\n\t"
        "v_readfirstlane_b32 %[tmp], v52\n\t"
        "s_add_u32 %[tmp], %[tmp], 1\n\t"
        "s_cmp_eq_u32 %[tmp], %[n]\n\t"
        "s_cbranch_scc1 2f\n\t"
        "s_bitcmp0_b32 %[spin], 23\n\t"
        "s_cbranch_scc1 1b\n\t"
        "s_branch 3f\n\t"
        "2:\n\t"
        "s_mov_b32 m0, %[m0]\n\t"
        "s_mov_b32 %[spin], 0\n\t"
        GL4("%[x]") NEXT4("%[x]") GL4("%[x]")
        "s_add_u32 m0, m0, 0x1000\n\ts_add_u32 %[x], %[x], 0x3f000\n\t"
        GL4("%[x]") NEXT4("%[x]") GL4("%[x]")
        "s_add_u32 m0, m0, 0x1000\n\ts_nop 0\n\t"
        GL4("%[w]") NEXT4("%[w]") GL4("%[w]") NEXT4("%[w]") GL4("%[w]") NEXT4("%[w]") GL4("%[w]")
        "v_mov_b32 v52, %[n]\n\t"
        "s_waitcnt vmcnt(0)\n\t"
        "ds_write_b32 %[afull], v52\n\t"
        "3:"
        : [x] "+&s"(x), [w] "+&s"(w), [tmp] "=&s"(tmp), [spin] "+&s"(spin)
        : [voff] "v"(lane * 16), [rs] "s"(rs), [acons] "v"(C_CONS + lw * 16), [afull] "v"(C_FULL + lw * 4), [m0] "s"(lw * SLOT), [n] "s"(n)
        : "scc", "memory", "v52", "v53", "v54", "v55");
    if (spin) return false;
  }
  return true;
}

#define TRY(k, n) "s_cmp_gt_u32 %[f" #k "], " n "\n\ts_cbranch_scc1 2" #k "f\n\t"
#define TAKE(k, n, base)                                         \
      "2" #k ":\n\t"                                             \
      "s_add_u32 " n ", " n ", 1\n\t"                            \
      "s_add_u32 %[xb], %[xoff], " base "\n\t"                   \
      "s_add_u32 %[wb], %[woff], " base "\n\t"                   \
      "v_add_u32 v50, " #k "*16, %[ac]\n\t"                      \
      "v_mov_b32 v51, " n "\n\t"
#define RDSTATE "ds_read_b128 v[52:55], v48\n\tds_read_b32 v49, v48 offset:16\n\t"

__global__ void __launch_bounds__(512, 2) gemm_ar_q8_fused(Params p) {
  extern __shared__ __attribute__((aligned(16))) char lds[];
  const u32 lane = threadIdx.x & 63, wid = rfl(threadIdx.x >> 6), b = blockIdx.x, rank = p.rank;
  if ((u32)(size_t)lds != 0) __builtin_trap();
  char* const myws = p.ws[rank];
  const u64 t0 = __builtin_amdgcn_s_memrealtime();
  const u32 seqv = __hip_atomic_load((u32*)(myws + OFF_CNT + b * 64), __ATOMIC_RELAXED, __HIP_MEMORY_SCOPE_AGENT);
  const u32 xg = b & 7, jb = b >> 3;
  if (wid == 0) STAMP(0);
  if (jb < NSJ) {
    const u32 sb = jb * 8 + xg, g = sb * SW + wid, d = AFF ? sb % 7 : g % 7, j = AFF ? (sb / 7) * SW + wid : g / 7;
    if (wid < SW && j < WR && ABL != 4) send_stream<true>(p, (rank + 1 + d) & 7, j, WR, OFF_CNT + b * 64, lane, t0);
    const u32 seq = rfl(seqv) + 1;
    if (DIRN < 8 && wid < SW3 && sb / 7 < NSJ * 8 / 7 && ABL != 4) send_out(p, (rank + 1 + d) & 7, (sb / 7) * SW3 + wid, seq, lane, t0);
    __syncthreads();
    if (threadIdx.x == 0) __hip_atomic_store((u32*)(myws + OFF_CNT + b * 64), seq, __ATOMIC_RELAXED, __HIP_MEMORY_SCOPE_AGENT);
    if (b == 0 && threadIdx.x < 8)
      __hip_atomic_store((u64*)(myws + OFF_Q + ((~seq & 1) * 8 + threadIdx.x) * 64), 0ull, __ATOMIC_RELAXED, __HIP_MEMORY_SCOPE_AGENT);
    return;
  }
  const u32 isB1 = jb >= NSJ + 16, i1 = isB1 ? jb - NSJ - 16 : jb - NSJ, njobs = (!isB1 && i1 < NSJ) ? 2 : 1, i2 = 16 - NSJ + i1;
  const u32 U1 = (xg & 1) * 4 + (i1 & 3), o1 = (xg >> 1) + 4 * (i1 >> 2), v1 = (o1 >> 1) + 8 * (o1 & 1);
  const u32 U2 = (xg & 1) * 4 + (i2 & 3), o2 = (xg >> 1) + 4 * (i2 >> 2), v2 = (o2 >> 1) + 8 * (o2 & 1);
  const u32 isB = isB1 || njobs == 2, UB = isB1 ? U1 : U2, vB = isB1 ? v1 : v2, tileB = UB * 16 + vB;
  if (PRET && isB1 && (vB & 7) >= 8 - DIRN && lane < 8) {
    const u32 a = __builtin_amdgcn_raw_buffer_load_b32(mk_rsrc(p.out[(wid + b + (b >> 3)) & 7], 16 * MiB), lane * 65536,
                                                       row0_of(rank, 2 * UB, 1) * 4096 + col0_of(rank, vB), AUX_SYS);
    asm volatile("" ::"v"(a));
  }
  if (wid == 0) lds_st32(CTRL + lane * 4, 0);
  asm volatile("s_waitcnt lgkmcnt(0)\n\ts_barrier" ::: "memory");
  const u32 wm = wid & 1, wn = (wid >> 1) & 1;
  float y[64];
#pragma unroll
  for (int i = 0; i < 64; ++i) y[i] = 0.f;

  if (wid < 4) {
   if (ABL != 1) {
    asm volatile("v_accvgpr_write_b32 a0, 0\n\tv_accvgpr_write_b32 a1, 0\n\tv_accvgpr_write_b32 a2, 0\n\tv_accvgpr_write_b32 a3, 0\n\tv_accvgpr_write_b32 a4, 0\n\tv_accvgpr_write_b32 a5, 0\n\tv_accvgpr_write_b32 a6, 0\n\tv_accvgpr_write_b32 a7, 0\n\tv_accvgpr_write_b32 a8, 0\n\tv_accvgpr_write_b32 a9, 0\n\tv_accvgpr_write_b32 a10, 0\n\tv_accvgpr_write_b32 a11, 0\n\tv_accvgpr_write_b32 a12, 0\n\tv_accvgpr_write_b32 a13, 0\n\tv_accvgpr_write_b32 a14, 0\n\tv_accvgpr_write_b32 a15, 0\n\tv_accvgpr_write_b32 a16, 0\n\tv_accvgpr_write_b32 a17, 0\n\tv_accvgpr_write_b32 a18, 0\n\tv_accvgpr_write_b32 a19, 0\n\tv_accvgpr_write_b32 a20, 0\n\tv_accvgpr_write_b32 a21, 0\n\tv_accvgpr_write_b32 a22, 0\n\tv_accvgpr_write_b32 a23, 0\n\tv_accvgpr_write_b32 a24, 0\n\tv_accvgpr_write_b32 a25, 0\n\tv_accvgpr_write_b32 a26, 0\n\tv_accvgpr_write_b32 a27, 0\n\tv_accvgpr_write_b32 a28, 0\n\tv_accvgpr_write_b32 a29, 0\n\tv_accvgpr_write_b32 a30, 0\n\tv_accvgpr_write_b32 a31, 0\n\tv_accvgpr_write_b32 a32, 0\n\tv_accvgpr_write_b32 a33, 0\n\tv_accvgpr_write_b32 a34, 0\n\tv_accvgpr_write_b32 a35, 0\n\tv_accvgpr_write_b32 a36, 0\n\tv_accvgpr_write_b32 a37, 0\n\tv_accvgpr_write_b32 a38, 0\n\tv_accvgpr_write_b32 a39, 0\n\tv_accvgpr_write_b32 a40, 0\n\tv_accvgpr_write_b32 a41, 0\n\tv_accvgpr_write_b32 a42, 0\n\tv_accvgpr_write_b32 a43, 0\n\tv_accvgpr_write_b32 a44, 0\n\tv_accvgpr_write_b32 a45, 0\n\tv_accvgpr_write_b32 a46, 0\n\tv_accvgpr_write_b32 a47, 0\n\tv_accvgpr_write_b32 a48, 0\n\tv_accvgpr_write_b32 a49, 0\n\tv_accvgpr_write_b32 a50, 0\n\tv_accvgpr_write_b32 a51, 0\n\tv_accvgpr_write_b32 a52, 0\n\tv_accvgpr_write_b32 a53, 0\n\tv_accvgpr_write_b32 a54, 0\n\tv_accvgpr_write_b32 a55, 0\n\tv_accvgpr_write_b32 a56, 0\n\tv_accvgpr_write_b32 a57, 0\n\tv_accvgpr_write_b32 a58, 0\n\tv_accvgpr_write_b32 a59, 0\n\tv_accvgpr_write_b32 a60, 0\n\tv_accvgpr_write_b32 a61, 0\n\tv_accvgpr_write_b32 a62, 0\n\tv_accvgpr_write_b32 a63, 0" ::: "a0", "a1", "a2", "a3", "a4", "a5", "a6", "a7", "a8", "a9", "a10", "a11", "a12", "a13", "a14", "a15", "a16", "a17", "a18", "a19", "a20", "a21", "a22", "a23", "a24", "a25", "a26", "a27", "a28", "a29", "a30", "a31", "a32", "a33", "a34", "a35", "a36", "a37", "a38", "a39", "a40", "a41", "a42", "a43", "a44", "a45", "a46", "a47", "a48", "a49", "a50", "a51", "a52", "a53", "a54", "a55", "a56", "a57", "a58", "a59", "a60", "a61", "a62", "a63");
    u32 fo[4];
    {
      const u32 i = lane & 31, h = lane >> 5;
#pragma unroll
      for (int s = 0; s < 4; ++s) fo[s] = i * 128 + (((2 * s + h) ^ ((i >> 1) & 7)) << 4);
    }
    const rsrc_t rs = mk_rsrc(myws, WS_USED);
    const u32 seq = rfl(seqv) + 1;
    u32 n0 = 0, n1 = 0, n2 = 0, n3 = 0, done = 0;
    const u32 total = 32 * njobs;
    u32 want = isB ? 3 : 1, stop = 32, inA = !isB1;
    for (;;) {
      u32 f0, f1, f2, f3, ctl, xb, wb;
      asm volatile(
      "v_mov_b32 v48, 0x20000\n\t"
      RDSTATE
      "1:\n\t"
      "s_waitcnt lgkmcnt(0)\n\t"
      "v_readfirstlane_b32 %[ctl], v49\n\t"
      "v_readfirstlane_b32 %[f0], v52\n\t"
      "v_readfirstlane_b32 %[f1], v53\n\t"
      "v_readfirstlane_b32 %[f2], v54\n\t"
      "v_readfirstlane_b32 %[f3], v55\n\t"
      "s_and_b32 %[xb], %[ctl], %[want]\n\t"
      "s_cbranch_scc1 9f\n\t"
      TRY(0, "%[n0]") TRY(1, "%[n1]") TRY(2, "%[n2]") TRY(3, "%[n3]")
      "s_sleep 1\n\t"
      RDSTATE
      "s_branch 1b\n\t"
      TAKE(0, "%[n0]", "0") "s_branch 6f\n\t"
      TAKE(1, "%[n1]", "0x8000") "s_branch 6f\n\t"
      TAKE(2, "%[n2]", "0x10000") "s_branch 6f\n\t"
      TAKE(3, "%[n3]", "0x18000")
      "6:\n\t"
v_accvgpr_read_b32 %0, a80\n\tv_accvgpr_read_b32 %1, a81\n\tv_accvgpr_read_b32 %2, a82\n\tv_accvgpr_read_b32 %3, a83\n\tv_accvgpr_read_b32 %4, a84\n\tv_accvgpr_read_b32 %5, a85\n\tv_accvgpr_read_b32 %6, a86\n\tv_accvgpr_read_b32 %7, a87\n\tv_accvgpr_read_b32 %8, a88\n\tv_accvgpr_read_b32 %9, a89\n\tv_accvgpr_read_b32 %10, a90\n\tv_accvgpr_read_b32 %11, a91\n\tv_accvgpr_read_b32 %12, a92\n\tv_accvgpr_read_b32 %13, a93\n\tv_accvgpr_read_b32 %14, a94\n\tv_accvgpr_read_b32 %15, a95      "s_add_u32 %[done], %[done], 1\n\t"
      "s_cmp_lt_u32 %[done], %[stop]\n\t"
      "s_cbranch_scc1 1b\n\t"
      "9:\n\t"
      "s_waitcnt lgkmcnt(0)\n\t"
      "s_nop 15"
      : [n0] "+&s"(n0), [n1] "+&s"(n1), [n2] "+&s"(n2), [n3] "+&s"(n3), [done] "+&s"(done),
        [f0] "=&s"(f0), [f1] "=&s"(f1), [f2] "=&s"(f2), [f3] "=&s"(f3), [ctl] "=&s"(ctl), [xb] "=&s"(xb), [wb] "=&s"(wb)
      : [fo0] "v"(fo[0]), [fo1] "v"(fo[1]), [fo2] "v"(fo[2]), [fo3] "v"(fo[3]), [ac] "v"(C_CONS + wid * 4),
        [want] "s"(want), [stop] "s"(stop), [xoff] "s"(wm * PIECE), [woff] "s"(2 * PIECE + wn * PIECE)
      : "scc", "memory", "v48", "v49", "v50", "v51", "v52", "v53", "v54", "v55", "v56", "v57", "v58", "v59", "v60", "v61", "v62", "v63", "v64", "v65", "v66", "v67", "v68", "v69", "v70", "v71", "v72", "v73", "v74", "v75", "v76", "v77", "v78", "v79", "v80", "v81", "v82", "v83", "v84", "v85", "v86", "v87", "v88", "v89", "v90", "v91", "v92", "v93", "v94", "v95", "v96", "v97", "v98", "v99", "v100", "v101", "v102", "v103", "v104", "v105", "v106", "v107", "v108", "v109", "v110", "v111", "v112", "v113", "v114", "v115", "v116", "v117", "v118", "v119", "v120", "v121", "v122", "v123", "v124", "v125", "v126", "v127", "a0", "a1", "a2", "a3", "a4", "a5", "a6", "a7", "a8", "a9", "a10", "a11", "a12", "a13", "a14", "a15", "a16", "a17", "a18", "a19", "a20", "a21", "a22", "a23", "a24", "a25", "a26", "a27", "a28", "a29", "a30", "a31", "a32", "a33", "a34", "a35", "a36", "a37", "a38", "a39", "a40", "a41", "a42", "a43", "a44", "a45", "a46", "a47", "a48", "a49", "a50", "a51", "a52", "a53", "a54", "a55", "a56", "a57", "a58", "a59", "a60", "a61", "a62", "a63");
      if (ctl & 1) break;
      if (inA && done == 32) {
        const u32 UA = U1, vA = v1;
        inA = 0;
        const u32 l = rank >> 2, l2 = 1 - l;
        const u32 qa = (l + (rank & 2) + (UA >> 2) - l2) & 3, qb = (l + 2 * (rank & 1) + (vA >> 3) - l2) & 3;
        const bool direct = (vA & 7) >= VD;
        const u32 ptile = ((qa & 1) * 4 + (UA & 3)) * 16 + (qb & 1) * 8 + (vA & 7), tileA = UA * 16 + vA;
        const rsrc_t rp = mk_rsrc(p.ws[direct ? l2 * 4 + (qa >> 1) * 2 + (qb >> 1) : rank], WS_USED);
        const u32 pdst = (direct ? OFF_P + ptile * 32768 : OFF_PO + tileA * 32768) + wid * PIECE;
        const u32 pflag = direct ? OFF_FP + ptile * 64 + wid * 4 : OFF_FP + tileA * 64 + 32 + wid * 4;
        const u32 pe = direct ? (lane >> 3) * 128 + (((lane & 7) ^ (lane >> 4)) << 4) : lane * 16, po = direct ? pe ^ 64 : pe;
#define PACK_STORE(J0)                                                                              \
        {                                                                                           \
          _Pragma("unroll") for (int j = 0; j < 2; ++j) {                                           \
            u32x4 o4;                                                                               \
            _Pragma("unroll") for (int e = 0; e < 4; ++e) {                                         \
              f32x2 f = {d[8 * j + 2 * e], d[8 * j + 2 * e + 1]};                                   \
              o4[e] = __builtin_bit_cast(u32, __builtin_convertvector(f, bf16x2));                  \
            }                                                                                       \
            st128(o4, rp, j ? po : pe, pdst + (J0 + j) * 1024);                                     \
          }                                                                                         \
        }
        { float d[16]; asm volatile("v_accvgpr_read_b32 %0, a0\n\tv_accvgpr_read_b32 %1, a1\n\tv_accvgpr_read_b32 %2, a2\n\tv_accvgpr_read_b32 %3, a3\n\tv_accvgpr_read_b32 %4, a4\n\tv_accvgpr_read_b32 %5, a5\n\tv_accvgpr_read_b32 %6, a6\n\tv_accvgpr_read_b32 %7, a7\n\tv_accvgpr_read_b32 %8, a8\n\tv_accvgpr_read_b32 %9, a9\n\tv_accvgpr_read_b32 %10, a10\n\tv_accvgpr_read_b32 %11, a11\n\tv_accvgpr_read_b32 %12, a12\n\tv_accvgpr_read_b32 %13, a13\n\tv_accvgpr_read_b32 %14, a14\n\tv_accvgpr_read_b32 %15, a15" : "=v"(d[0]), "=v"(d[1]), "=v"(d[2]), "=v"(d[3]), "=v"(d[4]), "=v"(d[5]), "=v"(d[6]), "=v"(d[7]), "=v"(d[8]), "=v"(d[9]), "=v"(d[10]), "=v"(d[11]), "=v"(d[12]), "=v"(d[13]), "=v"(d[14]), "=v"(d[15])); PACK_STORE(0) }
        { float d[16]; asm volatile("v_accvgpr_read_b32 %0, a16\n\tv_accvgpr_read_b32 %1, a17\n\tv_accvgpr_read_b32 %2, a18\n\tv_accvgpr_read_b32 %3, a19\n\tv_accvgpr_read_b32 %4, a20\n\tv_accvgpr_read_b32 %5, a21\n\tv_accvgpr_read_b32 %6, a22\n\tv_accvgpr_read_b32 %7, a23\n\tv_accvgpr_read_b32 %8, a24\n\tv_accvgpr_read_b32 %9, a25\n\tv_accvgpr_read_b32 %10, a26\n\tv_accvgpr_read_b32 %11, a27\n\tv_accvgpr_read_b32 %12, a28\n\tv_accvgpr_read_b32 %13, a29\n\tv_accvgpr_read_b32 %14, a30\n\tv_accvgpr_read_b32 %15, a31" : "=v"(d[0]), "=v"(d[1]), "=v"(d[2]), "=v"(d[3]), "=v"(d[4]), "=v"(d[5]), "=v"(d[6]), "=v"(d[7]), "=v"(d[8]), "=v"(d[9]), "=v"(d[10]), "=v"(d[11]), "=v"(d[12]), "=v"(d[13]), "=v"(d[14]), "=v"(d[15])); PACK_STORE(2) }
        { float d[16]; asm volatile("v_accvgpr_read_b32 %0, a32\n\tv_accvgpr_read_b32 %1, a33\n\tv_accvgpr_read_b32 %2, a34\n\tv_accvgpr_read_b32 %3, a35\n\tv_accvgpr_read_b32 %4, a36\n\tv_accvgpr_read_b32 %5, a37\n\tv_accvgpr_read_b32 %6, a38\n\tv_accvgpr_read_b32 %7, a39\n\tv_accvgpr_read_b32 %8, a40\n\tv_accvgpr_read_b32 %9, a41\n\tv_accvgpr_read_b32 %10, a42\n\tv_accvgpr_read_b32 %11, a43\n\tv_accvgpr_read_b32 %12, a44\n\tv_accvgpr_read_b32 %13, a45\n\tv_accvgpr_read_b32 %14, a46\n\tv_accvgpr_read_b32 %15, a47" : "=v"(d[0]), "=v"(d[1]), "=v"(d[2]), "=v"(d[3]), "=v"(d[4]), "=v"(d[5]), "=v"(d[6]), "=v"(d[7]), "=v"(d[8]), "=v"(d[9]), "=v"(d[10]), "=v"(d[11]), "=v"(d[12]), "=v"(d[13]), "=v"(d[14]), "=v"(d[15])); PACK_STORE(4) }
        { float d[16]; asm volatile("v_accvgpr_read_b32 %0, a48\n\tv_accvgpr_read_b32 %1, a49\n\tv_accvgpr_read_b32 %2, a50\n\tv_accvgpr_read_b32 %3, a51\n\tv_accvgpr_read_b32 %4, a52\n\tv_accvgpr_read_b32 %5, a53\n\tv_accvgpr_read_b32 %6, a54\n\tv_accvgpr_read_b32 %7, a55\n\tv_accvgpr_read_b32 %8, a56\n\tv_accvgpr_read_b32 %9, a57\n\tv_accvgpr_read_b32 %10, a58\n\tv_accvgpr_read_b32 %11, a59\n\tv_accvgpr_read_b32 %12, a60\n\tv_accvgpr_read_b32 %13, a61\n\tv_accvgpr_read_b32 %14, a62\n\tv_accvgpr_read_b32 %15, a63" : "=v"(d[0]), "=v"(d[1]), "=v"(d[2]), "=v"(d[3]), "=v"(d[4]), "=v"(d[5]), "=v"(d[6]), "=v"(d[7]), "=v"(d[8]), "=v"(d[9]), "=v"(d[10]), "=v"(d[11]), "=v"(d[12]), "=v"(d[13]), "=v"(d[14]), "=v"(d[15])); PACK_STORE(6) }
        asm volatile("v_accvgpr_write_b32 a0, 0\n\tv_accvgpr_write_b32 a1, 0\n\tv_accvgpr_write_b32 a2, 0\n\tv_accvgpr_write_b32 a3, 0\n\tv_accvgpr_write_b32 a4, 0\n\tv_accvgpr_write_b32 a5, 0\n\tv_accvgpr_write_b32 a6, 0\n\tv_accvgpr_write_b32 a7, 0\n\tv_accvgpr_write_b32 a8, 0\n\tv_accvgpr_write_b32 a9, 0\n\tv_accvgpr_write_b32 a10, 0\n\tv_accvgpr_write_b32 a11, 0\n\tv_accvgpr_write_b32 a12, 0\n\tv_accvgpr_write_b32 a13, 0\n\tv_accvgpr_write_b32 a14, 0\n\tv_accvgpr_write_b32 a15, 0\n\tv_accvgpr_write_b32 a16, 0\n\tv_accvgpr_write_b32 a17, 0\n\tv_accvgpr_write_b32 a18, 0\n\tv_accvgpr_write_b32 a19, 0\n\tv_accvgpr_write_b32 a20, 0\n\tv_accvgpr_write_b32 a21, 0\n\tv_accvgpr_write_b32 a22, 0\n\tv_accvgpr_write_b32 a23, 0\n\tv_accvgpr_write_b32 a24, 0\n\tv_accvgpr_write_b32 a25, 0\n\tv_accvgpr_write_b32 a26, 0\n\tv_accvgpr_write_b32 a27, 0\n\tv_accvgpr_write_b32 a28, 0\n\tv_accvgpr_write_b32 a29, 0\n\tv_accvgpr_write_b32 a30, 0\n\tv_accvgpr_write_b32 a31, 0\n\tv_accvgpr_write_b32 a32, 0\n\tv_accvgpr_write_b32 a33, 0\n\tv_accvgpr_write_b32 a34, 0\n\tv_accvgpr_write_b32 a35, 0\n\tv_accvgpr_write_b32 a36, 0\n\tv_accvgpr_write_b32 a37, 0\n\tv_accvgpr_write_b32 a38, 0\n\tv_accvgpr_write_b32 a39, 0\n\tv_accvgpr_write_b32 a40, 0\n\tv_accvgpr_write_b32 a41, 0\n\tv_accvgpr_write_b32 a42, 0\n\tv_accvgpr_write_b32 a43, 0\n\tv_accvgpr_write_b32 a44, 0\n\tv_accvgpr_write_b32 a45, 0\n\tv_accvgpr_write_b32 a46, 0\n\tv_accvgpr_write_b32 a47, 0\n\tv_accvgpr_write_b32 a48, 0\n\tv_accvgpr_write_b32 a49, 0\n\tv_accvgpr_write_b32 a50, 0\n\tv_accvgpr_write_b32 a51, 0\n\tv_accvgpr_write_b32 a52, 0\n\tv_accvgpr_write_b32 a53, 0\n\tv_accvgpr_write_b32 a54, 0\n\tv_accvgpr_write_b32 a55, 0\n\tv_accvgpr_write_b32 a56, 0\n\tv_accvgpr_write_b32 a57, 0\n\tv_accvgpr_write_b32 a58, 0\n\tv_accvgpr_write_b32 a59, 0\n\tv_accvgpr_write_b32 a60, 0\n\tv_accvgpr_write_b32 a61, 0\n\tv_accvgpr_write_b32 a62, 0\n\tv_accvgpr_write_b32 a63, 0" ::: "a0", "a1", "a2", "a3", "a4", "a5", "a6", "a7", "a8", "a9", "a10", "a11", "a12", "a13", "a14", "a15", "a16", "a17", "a18", "a19", "a20", "a21", "a22", "a23", "a24", "a25", "a26", "a27", "a28", "a29", "a30", "a31", "a32", "a33", "a34", "a35", "a36", "a37", "a38", "a39", "a40", "a41", "a42", "a43", "a44", "a45", "a46", "a47", "a48", "a49", "a50", "a51", "a52", "a53", "a54", "a55", "a56", "a57", "a58", "a59", "a60", "a61", "a62", "a63");
        asm volatile("s_waitcnt vmcnt(0)" ::: "memory");
        if (lane == 0) __builtin_amdgcn_raw_buffer_store_b32(seq, rp, 0, pflag, AUX_SYS);
        stop = total;
        if (wid == 0) STAMP(2);
      }
#define PLOAD                                                                                       \
        {                                                                                           \
          const u32 so = OFF_P + tileB * 32768 + wid * PIECE;                                       \
          const u32 xe = (lane >> 3) * 128 + (((lane & 7) ^ (lane >> 4)) << 4);                     \
          asm volatile("buffer_load_dwordx4 a[64:67], %0, %2, %3 offen offset:0 sc0 sc1\n\tbuffer_load_dwordx4 a[68:71], %1, %2, %3 offen offset:1024 sc0 sc1\n\tbuffer_load_dwordx4 a[72:75], %0, %2, %3 offen offset:2048 sc0 sc1\n\tbuffer_load_dwordx4 a[76:79], %1, %2, %3 offen offset:3072 sc0 sc1\n\tbuffer_load_dwordx4 a[80:83], %0, %2, %4 offen offset:0 sc0 sc1\n\tbuffer_load_dwordx4 a[84:87], %1, %2, %4 offen offset:1024 sc0 sc1\n\tbuffer_load_dwordx4 a[88:91], %0, %2, %4 offen offset:2048 sc0 sc1\n\tbuffer_load_dwordx4 a[92:95], %1, %2, %4 offen offset:3072 sc0 sc1" :: "v"(xe), "v"(xe ^ 64), "s"(rs), "s"(so), "s"(so + 4096) : "memory", "a64", "a65", "a66", "a67", "a68", "a69", "a70", "a71", "a72", "a73", "a74", "a75", "a76", "a77", "a78", "a79", "a80", "a81", "a82", "a83", "a84", "a85", "a86", "a87", "a88", "a89", "a90", "a91", "a92", "a93", "a94", "a95"); \
        }
      if (ctl & want & 2) {
        PLOAD
        want = 1;
      }
      if (done == total) {
        if (want == 1) break;
        if (CPOLL) {
          const u32* fp = (const u32*)(myws + OFF_FP + tileB * 64) + wid;
          while (__hip_atomic_load(fp, __ATOMIC_RELAXED, __HIP_MEMORY_SCOPE_SYSTEM) < seq)
            if (__builtin_amdgcn_s_memrealtime() - t0 > DEADLINE) { lds_st32(C_CTL, 1); break; }
          PLOAD
          break;
        }
        stop = total + 1;
      }
    }
    if (isB) {
      asm volatile("s_waitcnt vmcnt(0)" ::: "memory");
      { float* d = y; asm volatile("v_accvgpr_read_b32 %0, a0\n\tv_accvgpr_read_b32 %1, a1\n\tv_accvgpr_read_b32 %2, a2\n\tv_accvgpr_read_b32 %3, a3\n\tv_accvgpr_read_b32 %4, a4\n\tv_accvgpr_read_b32 %5, a5\n\tv_accvgpr_read_b32 %6, a6\n\tv_accvgpr_read_b32 %7, a7\n\tv_accvgpr_read_b32 %8, a8\n\tv_accvgpr_read_b32 %9, a9\n\tv_accvgpr_read_b32 %10, a10\n\tv_accvgpr_read_b32 %11, a11\n\tv_accvgpr_read_b32 %12, a12\n\tv_accvgpr_read_b32 %13, a13\n\tv_accvgpr_read_b32 %14, a14\n\tv_accvgpr_read_b32 %15, a15" : "=v"(d[0]), "=v"(d[1]), "=v"(d[2]), "=v"(d[3]), "=v"(d[4]), "=v"(d[5]), "=v"(d[6]), "=v"(d[7]), "=v"(d[8]), "=v"(d[9]), "=v"(d[10]), "=v"(d[11]), "=v"(d[12]), "=v"(d[13]), "=v"(d[14]), "=v"(d[15])); }
      { float* d = y + 16; asm volatile("v_accvgpr_read_b32 %0, a16\n\tv_accvgpr_read_b32 %1, a17\n\tv_accvgpr_read_b32 %2, a18\n\tv_accvgpr_read_b32 %3, a19\n\tv_accvgpr_read_b32 %4, a20\n\tv_accvgpr_read_b32 %5, a21\n\tv_accvgpr_read_b32 %6, a22\n\tv_accvgpr_read_b32 %7, a23\n\tv_accvgpr_read_b32 %8, a24\n\tv_accvgpr_read_b32 %9, a25\n\tv_accvgpr_read_b32 %10, a26\n\tv_accvgpr_read_b32 %11, a27\n\tv_accvgpr_read_b32 %12, a28\n\tv_accvgpr_read_b32 %13, a29\n\tv_accvgpr_read_b32 %14, a30\n\tv_accvgpr_read_b32 %15, a31" : "=v"(d[0]), "=v"(d[1]), "=v"(d[2]), "=v"(d[3]), "=v"(d[4]), "=v"(d[5]), "=v"(d[6]), "=v"(d[7]), "=v"(d[8]), "=v"(d[9]), "=v"(d[10]), "=v"(d[11]), "=v"(d[12]), "=v"(d[13]), "=v"(d[14]), "=v"(d[15])); }
      { float* d = y + 32; asm volatile("v_accvgpr_read_b32 %0, a32\n\tv_accvgpr_read_b32 %1, a33\n\tv_accvgpr_read_b32 %2, a34\n\tv_accvgpr_read_b32 %3, a35\n\tv_accvgpr_read_b32 %4, a36\n\tv_accvgpr_read_b32 %5, a37\n\tv_accvgpr_read_b32 %6, a38\n\tv_accvgpr_read_b32 %7, a39\n\tv_accvgpr_read_b32 %8, a40\n\tv_accvgpr_read_b32 %9, a41\n\tv_accvgpr_read_b32 %10, a42\n\tv_accvgpr_read_b32 %11, a43\n\tv_accvgpr_read_b32 %12, a44\n\tv_accvgpr_read_b32 %13, a45\n\tv_accvgpr_read_b32 %14, a46\n\tv_accvgpr_read_b32 %15, a47" : "=v"(d[0]), "=v"(d[1]), "=v"(d[2]), "=v"(d[3]), "=v"(d[4]), "=v"(d[5]), "=v"(d[6]), "=v"(d[7]), "=v"(d[8]), "=v"(d[9]), "=v"(d[10]), "=v"(d[11]), "=v"(d[12]), "=v"(d[13]), "=v"(d[14]), "=v"(d[15])); }
      { float* d = y + 48; asm volatile("v_accvgpr_read_b32 %0, a48\n\tv_accvgpr_read_b32 %1, a49\n\tv_accvgpr_read_b32 %2, a50\n\tv_accvgpr_read_b32 %3, a51\n\tv_accvgpr_read_b32 %4, a52\n\tv_accvgpr_read_b32 %5, a53\n\tv_accvgpr_read_b32 %6, a54\n\tv_accvgpr_read_b32 %7, a55\n\tv_accvgpr_read_b32 %8, a56\n\tv_accvgpr_read_b32 %9, a57\n\tv_accvgpr_read_b32 %10, a58\n\tv_accvgpr_read_b32 %11, a59\n\tv_accvgpr_read_b32 %12, a60\n\tv_accvgpr_read_b32 %13, a61\n\tv_accvgpr_read_b32 %14, a62\n\tv_accvgpr_read_b32 %15, a63" : "=v"(d[0]), "=v"(d[1]), "=v"(d[2]), "=v"(d[3]), "=v"(d[4]), "=v"(d[5]), "=v"(d[6]), "=v"(d[7]), "=v"(d[8]), "=v"(d[9]), "=v"(d[10]), "=v"(d[11]), "=v"(d[12]), "=v"(d[13]), "=v"(d[14]), "=v"(d[15])); }
      u32 mx = 0;
#define ADD_P(O)                                                                                    \
      _Pragma("unroll") for (int i = 0; i < 16; ++i) {                                              \
        y[O + 2 * i] += __uint_as_float(d[i] << 16);                                                \
        y[O + 2 * i + 1] += __uint_as_float(d[i] & 0xffff0000u);                                    \
        mx = max(mx, max(__float_as_uint(y[O + 2 * i]) & 0x7fffffffu, __float_as_uint(y[O + 2 * i + 1]) & 0x7fffffffu)); \
      }
      { u32 d[16]; asm volatile("v_accvgpr_read_b32 %0, a64\n\tv_accvgpr_read_b32 %1, a65\n\tv_accvgpr_read_b32 %2, a66\n\tv_accvgpr_read_b32 %3, a67\n\tv_accvgpr_read_b32 %4, a68\n\tv_accvgpr_read_b32 %5, a69\n\tv_accvgpr_read_b32 %6, a70\n\tv_accvgpr_read_b32 %7, a71\n\tv_accvgpr_read_b32 %8, a72\n\tv_accvgpr_read_b32 %9, a73\n\tv_accvgpr_read_b32 %10, a74\n\tv_accvgpr_read_b32 %11, a75\n\tv_accvgpr_read_b32 %12, a76\n\tv_accvgpr_read_b32 %13, a77\n\tv_accvgpr_read_b32 %14, a78\n\tv_accvgpr_read_b32 %15, a79" : "=v"(d[0]), "=v"(d[1]), "=v"(d[2]), "=v"(d[3]), "=v"(d[4]), "=v"(d[5]), "=v"(d[6]), "=v"(d[7]), "=v"(d[8]), "=v"(d[9]), "=v"(d[10]), "=v"(d[11]), "=v"(d[12]), "=v"(d[13]), "=v"(d[14]), "=v"(d[15])); ADD_P(0) }
      { u32 d[16]; asm volatile("v_accvgpr_read_b32 %0, a80\n\tv_accvgpr_read_b32 %1, a81\n\tv_accvgpr_read_b32 %2, a82\n\tv_accvgpr_read_b32 %3, a83\n\tv_accvgpr_read_b32 %4, a84\n\tv_accvgpr_read_b32 %5, a85\n\tv_accvgpr_read_b32 %6, a86\n\tv_accvgpr_read_b32 %7, a87\n\tv_accvgpr_read_b32 %8, a88\n\tv_accvgpr_read_b32 %9, a89\n\tv_accvgpr_read_b32 %10, a90\n\tv_accvgpr_read_b32 %11, a91\n\tv_accvgpr_read_b32 %12, a92\n\tv_accvgpr_read_b32 %13, a93\n\tv_accvgpr_read_b32 %14, a94\n\tv_accvgpr_read_b32 %15, a95" : "=v"(d[0]), "=v"(d[1]), "=v"(d[2]), "=v"(d[3]), "=v"(d[4]), "=v"(d[5]), "=v"(d[6]), "=v"(d[7]), "=v"(d[8]), "=v"(d[9]), "=v"(d[10]), "=v"(d[11]), "=v"(d[12]), "=v"(d[13]), "=v"(d[14]), "=v"(d[15])); ADD_P(32) }
      lds_st32(C_WMAX + wid * 4, wave_max(mx));
      if (wid == 0) STAMP(3);
    }
   }
  } else {
    const u32 lw = wid - 4;
    if (ABL != 4) send_stream<false>(p, rank, (b - 8 * NSJ) + (256 - 8 * NSJ) * lw, 4 * (256 - 8 * NSJ), OFF_CNT + b * 64, lane, t0);
    if (lw == 0) STAMP(7);
    const u32 seq = rfl(seqv) + 1;
    u32 n = 0;
    bool ok = true;
    if (ABL != 1) {
      ok = load_job(lw, isB1, U1, v1, n, myws, lane, seq, t0);
      if (njobs == 2) {
        for (u32 fin = !ok; !fin;) {
          u32 c;
          asm volatile("ds_read_b32 %0, %1\n\ts_waitcnt lgkmcnt(0)" : "=v"(c) : "v"(C_CONS + (lane & 15) * 4) : "memory");
          fin = __all(c >= 8);
          if (__builtin_amdgcn_s_memrealtime() - t0 > DEADLINE) { ok = false; fin = 1; }
        }
        if (ok) ok = load_job(lw, 1, U2, v2, n, myws, lane, seq, t0);
      }
      if (!ok) asm volatile("ds_or_b32 %0, %1" ::"v"(C_CTL), "v"(1u) : "memory");
    }
  }
  asm volatile("s_waitcnt lgkmcnt(0)" ::: "memory");
  __syncthreads();
  const u32 seq = rfl(seqv) + 1;
  if (threadIdx.x == 0) __hip_atomic_store((u32*)(myws + OFF_CNT + b * 64), seq, __ATOMIC_RELAXED, __HIP_MEMORY_SCOPE_AGENT);
  if (!isB) return;
  if (ABL == 4) {
    if (wid < 4 && y[0] == 123.456f) p.out[0][0] = 1;
    return;
  }
  {
    const u32x4 wmx = *(const u32x4*)(lds + C_WMAX);
    const u32 m = max(max(wmx[0], wmx[1]), max(wmx[2], wmx[3]));
    if (lane == 0)
      __hip_atomic_store((u64*)(p.ws[wid] + OFF_G) + rank * 128 + tileB, ((u64)seq << 32) | m, __ATOMIC_RELAXED, __HIP_MEMORY_SCOPE_SYSTEM);
    const rsrc_t rs = mk_rsrc(myws, WS_USED);
    u32 gm;
    for (;;) {
      const u32x4 g0 = __builtin_amdgcn_raw_buffer_load_b128(rs, lane * 16, OFF_G + wid * 1024, GAUX);
      gm = max(g0[0], g0[2]);
      if (__all(g0[1] == seq && g0[3] == seq)) break;
      if (__builtin_amdgcn_s_memrealtime() - t0 > DEADLINE) { lds_st32(C_CTL, 1); break; }
      if (GSLEEP) __builtin_amdgcn_s_sleep(GSLEEP);
      asm volatile("" ::: "memory");
    }
    lds_st32(C_GMAX + wid * 4, wave_max(gm));
  }
  asm volatile("s_waitcnt lgkmcnt(0)" ::: "memory");
  __syncthreads();
  if (wid == 0) STAMP(4);
  if (wid < 4) {
    const u32x4 ga = *(const u32x4*)(lds + C_GMAX), gb = *(const u32x4*)(lds + C_GMAX + 16);
    const u32 ab0 = *(const u32*)(lds + C_CTL) & 1;
    const float amax = __uint_as_float(max(max(max(ga[0], ga[1]), max(ga[2], ga[3])), max(max(gb[0], gb[1]), max(gb[2], gb[3]))));
    const float scale = amax / 448.0f;
    const bool ab = ab0 || amax != amax;
    const float rinv = scale > 0.f || scale != scale ? 1.0f / scale : 0.f;
    const u32 h = lane >> 5;
#pragma unroll
    for (int t = 0; t < 4; ++t) {
      u32x4 o4;
#pragma unroll
      for (int e = 0; e < 4; ++e) {
        const float* yy = y + 16 * t + 4 * e;
        int pk = __builtin_amdgcn_cvt_pk_fp8_f32(yy[0] * rinv, yy[1] * rinv, 0, false);
        pk = __builtin_amdgcn_cvt_pk_fp8_f32(yy[2] * rinv, yy[3] * rinv, pk, true);
        o4[e] = ab ? 0x7f7f7f7fu : (u32)pk;
      }
      const u32 m = wm * 64 + (t >> 1) * 32 + (lane & 31), ch = 4 * wn + 2 * (t & 1) + h;
      *(u32x4*)(lds + m * 128 + ((ch ^ (m & 7)) << 4)) = o4;
    }
  }
  __syncthreads();
  if (wid == 0) STAMP(1);
  {
    const u32 pq = (wid + b + (b >> 3)) & 7;
    const u32 so = row0_of(rank, 2 * UB, 1) * 4096 + col0_of(rank, vB);
    const u32 r8 = lane >> 3, c = lane & 7;
    if ((vB & 7) < 8 - DIRN) {
      const rsrc_t rm = mk_rsrc(p.out[rank], 16 * MiB);
#pragma unroll
      for (int t = 0; t < 2; ++t) {
        const u32 row = 16 * wid + 8 * t + r8;
        st128(*(const u32x4*)(lds + row * 128 + ((c ^ r8) << 4)), rm, row * 4096 + c * 16, so);
      }
      asm volatile("s_waitcnt vmcnt(0)\n\ts_barrier" ::: "memory");
      if (threadIdx.x == 0) __hip_atomic_store((u32*)(myws + OFF_FT) + tileB, seq, __ATOMIC_RELAXED, __HIP_MEMORY_SCOPE_SYSTEM);
      return;
    }
    const rsrc_t ro = mk_rsrc(p.out[pq], 16 * MiB);
#pragma unroll
    for (int t = 0; t < 16; ++t) {
      const u32 row = 8 * ((t + 2 * pq + b) & 15) + r8;
      const u32x4 o4 = *(const u32x4*)(lds + row * 128 + ((c ^ (row & 7)) << 4));
      st128(o4, ro, row * 4096 + c * 16, so);
    }
    asm volatile("s_waitcnt vmcnt(0)" ::: "memory");
    if (wid == 0) STAMP(5);
    if (pq != rank) {
      if (lane == 0)
        __hip_atomic_store((u32*)(p.ws[pq] + OFF_FD + tileB * 64) + rank, seq, __ATOMIC_RELAXED, __HIP_MEMORY_SCOPE_SYSTEM);
      const u32* fd = (const u32*)(myws + OFF_FD + tileB * 64) + pq;
      while (__hip_atomic_load(fd, __ATOMIC_RELAXED, __HIP_MEMORY_SCOPE_SYSTEM) < seq) {
        if (__builtin_amdgcn_s_memrealtime() - t0 > DEADLINE) {
          if (lane == 0) p.out[rank][so] = 0x7f;
          break;
        }
        __builtin_amdgcn_s_sleep(8);
      }
    }
    if (pq == ((rank + 1) & 7)) STAMP(6);
  }
}

extern "C" void kernel_launch(int rank, int nranks, ncclComm_t comm, hipStream_t stream, void* const* d_in,
                              const int* in_sizes, int n_in, void* d_out, int out_size, const int* in_dtypes,
                              int out_dtype, void* const* d_out_all, void* const* d_ws_all, size_t ws_bytes) {
  if (nranks != 8 || ws_bytes < WS_USED || (unsigned)rank >= 8u) {
    fprintf(stderr, "unsupported configuration\n");
    return;
  }
  (void)hipFuncSetAttribute((const void*)gemm_ar_q8_fused, hipFuncAttributeMaxDynamicSharedMemorySize, LDS_BYTES);
  Params p;
  p.x = d_in[0];
  p.w = d_in[1];
  for (int i = 0; i < 8; ++i) {
    p.ws[i] = (char*)d_ws_all[i];
    p.out[i] = (unsigned char*)d_out_all[i];
  }
  p.rank = rank;
  gemm_ar_q8_fused<<<256, 512, LDS_BYTES, stream>>>(p);
}
```

```cpp
#include <hip/hip_runtime.h>
#include <rccl/rccl.h>
#include <cstdio>
#include <cstdint>

typedef unsigned int u32;
typedef unsigned long long u64;
typedef u32 u32x4 __attribute__((ext_vector_type(4)));
typedef u32 u32x2 __attribute__((ext_vector_type(2)));
typedef __bf16 bf16x2 __attribute__((ext_vector_type(2)));
typedef float f32x2 __attribute__((ext_vector_type(2)));
typedef float f32x4 __attribute__((ext_vector_type(4)));
typedef __amdgpu_buffer_rsrc_t rsrc_t;

#define NSJ 5
#define SW 2
#define WR 9
#define WR3 (NSJ * SW)
#define DIRN 2
static_assert(WR <= WR3 && DIRN >= 1 && DIRN < 8, "sender layout");

constexpr u32 MiB = 1u << 20;
constexpr u32 PIECE = 8192;
constexpr u32 OFF_XS = 0;
constexpr u32 OFF_XO = 4 * MiB;
constexpr u32 OFF_W = 8 * MiB;
constexpr u32 OFF_P = 16 * MiB;
constexpr u32 OFF_PO = 20 * MiB;
constexpr u32 OFF_FX = 24 * MiB;
constexpr u32 OFF_FW = OFF_FX + 4096;
constexpr u32 OFF_FP = OFF_FW + 4096;
constexpr u32 OFF_G = OFF_FP + 8192;
constexpr u32 OFF_FD = OFF_G + 8192;
constexpr u32 OFF_FT = OFF_FD + 8192;
constexpr u32 OFF_FE = OFF_FT + 512;
constexpr u32 OFF_CNT = OFF_FE + 1024 + 512;
constexpr u32 OFF_Q = OFF_CNT + 16384;
constexpr u32 OFF_R = OFF_Q + 1024;
constexpr u32 OFF_DBG = OFF_R + 1024;
constexpr u32 WS_USED = OFF_DBG + 256 * 128 * 2;

constexpr u32 SLOT = 4 * PIECE;
constexpr u32 CTRL = 4 * SLOT;
constexpr u32 LDS_BYTES = CTRL + 256;
constexpr u32 C_FULL = CTRL + 0;
constexpr u32 C_CTL = CTRL + 16;
constexpr u32 C_CONS = CTRL + 32;
constexpr u32 C_WMAX = CTRL + 96;
constexpr u32 C_GMAX = CTRL + 128;
static_assert(C_FULL == 0x20000 && SLOT == 0x8000, "literals in the asm loops");
constexpr u64 DEADLINE = 100000000ull;
constexpr int AUX_SYS = 17;

struct Params {
  const void* x;
  const void* w;
  char* ws[8];
  unsigned char* out[8];
  int rank;
};

__device__ __forceinline__ u32 rfl(u32 v) { return __builtin_amdgcn_readfirstlane(v); }
__device__ __forceinline__ rsrc_t mk_rsrc(const void* p, u32 bytes) {
  return __builtin_amdgcn_make_buffer_rsrc((void*)p, (short)0, (int)bytes, 0x00020000);
}
__device__ __forceinline__ void lds_st32(u32 addr, u32 v) { asm volatile("ds_write_b32 %0, %1" ::"v"(addr), "v"(v) : "memory"); }
__device__ __forceinline__ void st128(u32x4 v, rsrc_t r, u32 voff, u32 soff) {
  asm volatile("buffer_store_dwordx4 %0, %1, %2, %3 offen sc0 sc1\n\ts_nop 1" ::"v"(v), "v"(voff), "s"(r), "s"(soff) : "memory");
}
template <int OFF> __device__ __forceinline__ void st128i(u32x4 v, rsrc_t r, u32 voff, u32 soff) {
  asm volatile("buffer_store_dwordx4 %0, %1, %2, %3 offen offset:%4 sc0 sc1\n\ts_nop 1" ::"v"(v), "v"(voff), "s"(r), "s"(soff), "i"(OFF) : "memory");
}
__device__ __forceinline__ u32 wave_max(u32 x) {
  x = max(x, (u32)__builtin_amdgcn_update_dpp(0, (int)x, 0xB1, 0xf, 0xf, true));
  x = max(x, (u32)__builtin_amdgcn_update_dpp(0, (int)x, 0x4E, 0xf, 0xf, true));
  x = max(x, (u32)__builtin_amdgcn_update_dpp(0, (int)x, 0x141, 0xf, 0xf, true));
  x = max(x, (u32)__builtin_amdgcn_update_dpp(0, (int)x, 0x140, 0xf, 0xf, true));
  return max(max((u32)__builtin_amdgcn_readlane((int)x, 0), (u32)__builtin_amdgcn_readlane((int)x, 16)),
             max((u32)__builtin_amdgcn_readlane((int)x, 32), (u32)__builtin_amdgcn_readlane((int)x, 48)));
}
__device__ __forceinline__ void sgpr_pad(u32 s) { asm volatile("s_nop 4" ::"s"(s)); }
#define SSTAMP(off) do { } while (0)
#define STAMP(i) do { } while (0)

__device__ __forceinline__ u32 row0_of(u32 r, u32 u, u32 own) {
  const u32 l = r >> 2, i = (r >> 1) & 1;
  const u32 quad = (l + 2 * i + (u >> 3)) & 3;
  const u32 half = own ? l : 1 - l;
  return quad * 1024 + half * 512 + (u & 7) * 64;
}
__device__ __forceinline__ u32 col0_of(u32 r, u32 v) {
  const u32 l = r >> 2, j = r & 1;
  return ((l + 2 * j + (v >> 3)) & 3) * 1024 + (v & 7) * 128;
}

__constant__ const u32 w_order[80] = {0, 8, 1, 16, 9, 2, 24, 17, 10, 32, 3, 25, 18, 40, 11, 33, 4, 26, 48, 19, 41, 12, 34, 56, 27, 49, 20, 42, 64, 35, 57, 28, 50, 72, 43, 65, 36, 58, 80, 51, 73, 44, 66, 88, 59, 81, 52, 74, 96, 67, 89, 60, 82, 104, 75, 97, 68, 90, 112, 83, 105, 76, 98, 120, 91, 113, 84, 106, 99, 121, 92, 114, 107, 100, 122, 115, 108, 123, 116, 124};
struct PieceD { u32 kind, src, dst, flag, dep; };
__device__ __forceinline__ PieceD decode(u32 q, u32 rank, u32 dest, u32 nk, u32 kbase, u32 step0) {
  PieceD d;
  const u32 nx = 16 * nk;
  d.dep = OFF_FP;
  if (q >= 4 * nx) {
    const u32 qq = q - 4 * nx, l = rank >> 2, l2 = 1 - l;
    const u32 qa0 = (l + (rank & 2) - l2) & 3, qb0 = (l + 2 * (rank & 1) - l2) & 3;
    const u32 uh = (qa0 >> 1) != ((dest >> 1) & 1), vh = (qb0 >> 1) != (dest & 1);
    const u32 qa = (qa0 + uh) & 3, qb = (qb0 + vh) & 3;
    const u32 ul = (qq >> 2) & 3, vl = qq >> 4, h = qq & 3;
    const u32 tile = (uh * 4 + ul) * 16 + vh * 8 + vl, ptile = ((qa & 1) * 4 + ul) * 16 + (qb & 1) * 8 + vl;
    d.kind = 2;
    d.src = OFF_PO + tile * 32768 + h * PIECE;
    d.dst = OFF_P + ptile * 32768 + h * PIECE;
    d.flag = OFF_FP + ptile * 64 + h * 4;
    d.dep = OFF_FP + tile * 64 + 32 + h * 4;
  } else if (q >= nx && q < 3 * nx) {
    const u32 qq = q - nx;
    const u32 e5 = w_order[rfl(min(qq >> 1, 79u))];
    const u32 o = (nk == 5) ? e5 >> 3 : qq / 6, ks = (nk == 5) ? e5 & 7 : (qq - o * 6) >> 1;
    const u32 half = qq & 1, v = (o >> 1) + 8 * (o & 1), step = step0 + ks;
    d.kind = 1;
    d.src = (kbase + 64 * ks) * 8192 + (col0_of(dest, v) + 64 * half) * 2;
    d.dst = OFF_W + ((v * 32 + step) * 2 + half) * PIECE;
    d.flag = OFF_FW + ((2 * v + half) * 32 + step) * 4;
  } else {
    const u32 own = q >= nx, qq = own ? q - 3 * nx : q;
    const u32 ks = qq >> 4, u = qq & 15, step = step0 + ks;
    d.kind = 0;
    d.src = row0_of(dest, u, own) * 1024 + (kbase + 64 * ks) * 2;
    d.dst = (own ? OFF_XO : OFF_XS) + (u * 32 + step) * PIECE;
    d.flag = OFF_FX + ((own * 16 + u) * 32 + step) * 4;
  }
  d.kind = rfl(d.kind); d.src = rfl(d.src); d.dst = rfl(d.dst); d.flag = rfl(d.flag); d.dep = rfl(d.dep);
  return d;
}

__device__ __forceinline__ void piece_load(u32x4 (&v)[8], const PieceD& d, rsrc_t rx, rsrc_t rw, rsrc_t rl, u32 lane) {
  const u32 kind = rfl(d.kind);
  const u32 vo = (lane >> 3) * (kind == 1 ? 65536u : kind == 0 ? 1024u : 128u) + (lane & 7) * 16;
  const rsrc_t r = kind == 1 ? rw : kind == 0 ? rx : rl;
  const u32 st = rfl(kind == 2 ? 1024u : 8192u);
  u32 so = rfl(d.src);
  asm volatile("s_nop 4\n\tbuffer_load_dwordx4 %0, %9, %10, %8 offen sc1\n\ts_add_u32 %8, %8, %11\n\t"
               "buffer_load_dwordx4 %1, %9, %10, %8 offen sc1\n\ts_add_u32 %8, %8, %11\n\t"
               "buffer_load_dwordx4 %2, %9, %10, %8 offen sc1\n\ts_add_u32 %8, %8, %11\n\t"
               "buffer_load_dwordx4 %3, %9, %10, %8 offen sc1\n\ts_add_u32 %8, %8, %11\n\t"
               "buffer_load_dwordx4 %4, %9, %10, %8 offen sc1\n\ts_add_u32 %8, %8, %11\n\t"
               "buffer_load_dwordx4 %5, %9, %10, %8 offen sc1\n\ts_add_u32 %8, %8, %11\n\t"
               "buffer_load_dwordx4 %6, %9, %10, %8 offen sc1\n\ts_add_u32 %8, %8, %11\n\t"
               "buffer_load_dwordx4 %7, %9, %10, %8 offen sc1"
               : "=&v"(v[0]), "=&v"(v[1]), "=&v"(v[2]), "=&v"(v[3]), "=&v"(v[4]), "=&v"(v[5]), "=&v"(v[6]), "=&v"(v[7]), "+&s"(so)
               : "v"(vo), "s"(r), "s"(st) : "scc", "memory");
}
__device__ __forceinline__ void dep_load(u32& f, u32 dep, rsrc_t rl) {
  asm volatile("s_nop 4\n\tbuffer_load_dword %0, off, %1, %2 sc0 sc1" : "=&v"(f) : "s"(rl), "s"(rfl(dep)) : "memory");
}
__device__ __forceinline__ void claim(u32& c, u32 qoff, rsrc_t rl) {
  c = 1;
  asm volatile("s_mov_b64 exec, 1\n\tbuffer_atomic_add %0, off, %1, %2 sc0\n\ts_mov_b64 exec, -1" : "+v"(c) : "s"(rl), "s"(qoff) : "memory");
}
__device__ __forceinline__ void ticket(u32& c, u32 seq, u32 toff, rsrc_t rl) {
  c = seq;
  asm volatile("s_nop 3\n\ts_mov_b64 exec, 1\n\tbuffer_atomic_swap %0, off, %1, %2 sc0\n\ts_mov_b64 exec, -1" : "+v"(c) : "s"(rl), "s"(rfl(toff)) : "memory");
}
#define WAITV(N, v, f, c) asm volatile("s_waitcnt vmcnt(" #N ")" : "+v"(v[0]), "+v"(v[1]), "+v"(v[2]), "+v"(v[3]), "+v"(v[4]), "+v"(v[5]), "+v"(v[6]), "+v"(v[7]), "+v"(f), "+v"(c) :: "memory")
template <int E> __device__ __forceinline__ void w_store(const u32x4 (&v)[8], rsrc_t rd, u32 wo, u32 dst) {
  u32x4 o;
#pragma unroll
  for (int j = 0; j < 4; ++j)
    o[j] = __builtin_amdgcn_perm(v[2 * j + 1][E >> 1], v[2 * j][E >> 1], (E & 1) ? 0x07060302u : 0x05040100u);
  st128i<((E & 3) + 8 * (E >> 2)) * 128>(o, rd, wo ^ ((((E >> 1) & 1) + 4 * (E >> 2)) << 4), dst);
}
__device__ __forceinline__ void piece_store(const u32x4 (&v)[8], const PieceD& d, rsrc_t rd, u32 lane) {
  const u32 hi = lane >> 3, lo = lane & 7, dst = rfl(d.dst), kind = rfl(d.kind);
  sgpr_pad(dst);
  if (kind == 1) {
    const u32 wo = (32 * (lo >> 2) + 16 * (lo & 1) + 4 * ((lo >> 1) & 1)) * 128 + ((hi ^ (lo & 2)) << 4);
    w_store<0>(v, rd, wo, dst); w_store<1>(v, rd, wo, dst); w_store<2>(v, rd, wo, dst); w_store<3>(v, rd, wo, dst);
    w_store<4>(v, rd, wo, dst); w_store<5>(v, rd, wo, dst); w_store<6>(v, rd, wo, dst); w_store<7>(v, rd, wo, dst);
  } else {
    const u32 xe = hi * 128 + ((lo ^ (hi >> 1)) << 4), xo = xe ^ 64, dst2 = dst + 4096;
    st128i<0>(v[0], rd, xe, dst); st128i<1024>(v[1], rd, xo, dst); st128i<2048>(v[2], rd, xe, dst); st128i<3072>(v[3], rd, xo, dst);
    st128i<0>(v[4], rd, xe, dst2); st128i<1024>(v[5], rd, xo, dst2); st128i<2048>(v[6], rd, xe, dst2); st128i<3072>(v[7], rd, xo, dst2);
  }
}

template <bool DYN>
__device__ __forceinline__ bool send_stream(const Params& p, u32 dest, u32 q0, u32 qs, u32 cntoff, u32 toff, u32 lane, u64 t0) {
  const u32 rank = p.rank;
  const bool same = (dest >> 2) == (rank >> 2);
  const u32 nk = same ? 5 : 3, kbase = same ? 0 : 320, step0 = same ? (rank & 3) * 5 : 20 + (rank & 3) * 3;
  const u32 total = 320;
  if (q0 >= total) return false;
  const rsrc_t rx = mk_rsrc(p.x, 4 * MiB), rw = mk_rsrc(p.w, 4 * MiB), rd = mk_rsrc(p.ws[dest], WS_USED), rl = mk_rsrc(p.ws[rank], WS_USED);
  u32x4 bufA[8], bufB[8];
  u32 f = 0;
  u32 cl = 0, i1 = q0 + qs, i2 = q0 + 2 * qs;
  PieceD dc = decode(q0, rank, dest, nk, kbase, step0);
  dep_load(f, cntoff, rl);
  piece_load(bufA, dc, rx, rw, rl, lane);
  PieceD dn = decode(min(i1, total - 1), rank, dest, nk, kbase, step0);
  piece_load(bufB, dn, rx, rw, rl, lane);
  WAITV(8, bufA, f, cl);
  const u32 seq = rfl(f) + 1;
  const u32 qoff = OFF_Q + ((seq & 1) * 8 + dest) * 64;
  u32 prevflag = 0;
  bool rdyn = true;
#define FLAG(off) __builtin_amdgcn_raw_buffer_store_b32(seq, rd, 0, rfl(off), AUX_SYS)
#define SEND_HALF(cur, nxt, LD)                                                               \
  {                                                                                           \
    const bool more = i1 < total;                                                             \
    const PieceD d2 = decode(min(i2, total - 1), rank, dest, nk, kbase, step0);               \
    if (more && !rdyn) {                          \
      u32 g;                                                                                  \
      dep_load(g, dn.dep, rl);                                                                \
      asm volatile("s_waitcnt vmcnt(0)" : "+v"(g) :: "memory");                               \
      if (prevflag && lane == 0) FLAG(prevflag);                                              \
      prevflag = 0;                                                                           \
      rdyn = rfl(g) >= seq;                                                                   \
    }                                                                                         \
    const u32 slow = rfl(more && !rdyn);                                                      \
    if (slow) {                                                                               \
      piece_store(cur, dc, rd, lane);                                                         \
      asm volatile("s_waitcnt vmcnt(0)" ::: "memory");                                        \
      if (lane == 0) { if (prevflag) FLAG(prevflag); FLAG(dc.flag); }                         \
      prevflag = 0;                                                                           \
      for (;;) {                                                                              \
        u32 g;                                                                                \
        dep_load(g, dn.dep, rl);                                                              \
        asm volatile("s_waitcnt vmcnt(0)" : "+v"(g) :: "memory");                             \
        if (rfl(g) >= seq) break;                                                             \
        if (__builtin_amdgcn_s_memrealtime() - t0 > DEADLINE) return false;     \
        __builtin_amdgcn_s_sleep(1);                                                          \
      }                                                                                       \
    }                                                                                         \
    if (DYN) { if (LD) claim(cl, qoff, rl); else ticket(cl, seq, toff, rl); }                 \
    dep_load(f, d2.dep, rl);                                                                  \
    if (LD) piece_load(nxt, dn, rx, rw, rl, lane);         \
    if (!slow) piece_store(cur, dc, rd, lane);                                                \
    else asm volatile("s_waitcnt vmcnt(0)" ::: "memory");                                     \
    WAITV(8, nxt, f, cl);                                                                      \
    if (prevflag && lane == 0) FLAG(prevflag);                                                \
    prevflag = slow ? 0u : dc.flag;                                                           \
    if (!more) break;                                                                         \
    rdyn = d2.kind != 2 || rfl(f) >= seq;                                       \
    dc = dn; dn = d2;                                                                         \
    if (DYN && !LD && rfl(cl) == seq) {           \
      asm volatile("s_waitcnt vmcnt(0)" ::: "memory");                                        \
      return true;                                                                            \
    }                                                                                         \
    i1 = i2; i2 = (DYN && LD) ? 4 * qs + rfl(cl) : i2 + qs;                                   \
  }
  for (;;) {
    SEND_HALF(bufA, bufB, false)
    for (;;) {
      SEND_HALF(bufB, bufA, true)
      SEND_HALF(bufA, bufB, true)
    }
    break;
  }
  asm volatile("s_waitcnt vmcnt(0)" ::: "memory");
  if (prevflag && lane == 0) __builtin_amdgcn_raw_buffer_store_b32(seq, rd, 0, rfl(prevflag), AUX_SYS);
  return false;
}

__device__ __forceinline__ u32 out_off(u32 q, u32 rank, u32 rot) {
  constexpr u32 NB = 8 - DIRN, NP = 8 * 2 * NB * 2;
  u32 r = min(q, NP - 1) + rot;
  r = r >= NP ? r - NP : r;
  const u32 tl = r >> 1, U = tl / (2 * NB), vv = tl - U * (2 * NB), v = vv < NB ? vv : vv + DIRN;
  return rfl(row0_of(rank, 2 * U + (r & 1), 1) * 4096 + col0_of(rank, v));
}
#define OUT8(op, v, r, so)                                                                    \
  asm volatile("s_nop 4\n\t" op " %0, %9, %10, %8 offen sc0 sc1\n\ts_add_u32 %8, %8, 0x8000\n\t"           \
               op " %1, %9, %10, %8 offen sc0 sc1\n\ts_add_u32 %8, %8, 0x8000\n\t"                         \
               op " %2, %9, %10, %8 offen sc0 sc1\n\ts_add_u32 %8, %8, 0x8000\n\t"                         \
               op " %3, %9, %10, %8 offen sc0 sc1\n\ts_add_u32 %8, %8, 0x8000\n\t"                         \
               op " %4, %9, %10, %8 offen sc0 sc1\n\ts_add_u32 %8, %8, 0x8000\n\t"                         \
               op " %5, %9, %10, %8 offen sc0 sc1\n\ts_add_u32 %8, %8, 0x8000\n\t"                         \
               op " %6, %9, %10, %8 offen sc0 sc1\n\ts_add_u32 %8, %8, 0x8000\n\t"                         \
               op " %7, %9, %10, %8 offen sc0 sc1\n\ts_nop 1"                                               \
               : "+v"(v[0]), "+v"(v[1]), "+v"(v[2]), "+v"(v[3]), "+v"(v[4]), "+v"(v[5]), "+v"(v[6]), "+v"(v[7]), "+&s"(so) \
               : "v"(vo), "s"(r) : "scc", "memory")
__device__ __forceinline__ void send_out(const Params& p, u32 dest, u32 j, u32 seq, u32 lane, u64 t0) {
  constexpr u32 NP = 8 * 2 * (8 - DIRN) * 2;
  const u32 rank = p.rank;
  const rsrc_t ro = mk_rsrc(p.out[rank], 16 * MiB), rd = mk_rsrc(p.out[dest], 16 * MiB), rl = mk_rsrc(p.ws[rank], WS_USED);
  const u32 qoff = OFF_Q + ((seq & 1) * 8 + dest) * 64 + 4, rot = ((rank * 8 + dest) * 37) % NP;
  const u32 vo = (lane >> 3) * 4096 + (lane & 7) * 16;
  bool ok = true;
  for (;;) {
    const u32x2 f = __builtin_amdgcn_raw_buffer_load_b64(rl, lane * 8, OFF_FT, AUX_SYS);
    const bool skip = ((2 * lane) & 7) >= 8 - DIRN, skip1 = ((2 * lane + 1) & 7) >= 8 - DIRN;
    if (__all((skip || f[0] >= seq) && (skip1 || f[1] >= seq))) break;
    if (__builtin_amdgcn_s_memrealtime() - t0 > DEADLINE) {
      if (lane == 0) p.out[rank][out_off(j, rank, rot)] = 0x7f;
      return;
    }
    asm volatile("" ::: "memory");
  }
  u32x4 bufA[8] = {}, bufB[8] = {};
  u32 cl = 0, dummy = 0, i1 = j + WR3, oc = out_off(j, rank, rot);
  { u32 so = oc; OUT8("buffer_load_dwordx4", bufA, ro, so); }
  { u32 so = out_off(i1, rank, rot); OUT8("buffer_load_dwordx4", bufB, ro, so); }
  WAITV(8, bufA, dummy, cl);
#define OUT_HALF(cur, nxt, LD)                                                                \
  {                                                                                           \
    const u32 on = out_off(i1, rank, rot);                                                    \
    claim(cl, qoff, rl);                                                                      \
    if (LD) { u32 so = on; OUT8("buffer_load_dwordx4", nxt, ro, so); }                        \
    { u32 so = oc; OUT8("buffer_store_dwordx4", cur, rd, so); }                               \
    WAITV(8, nxt, dummy, cl);                                                                 \
    if (i1 >= NP) break;                                                                      \
    oc = on; i1 = 2 * WR3 + rfl(cl);                                                          \
  }
  for (;;) {
    OUT_HALF(bufA, bufB, false)
    for (;;) {
      OUT_HALF(bufB, bufA, true)
      OUT_HALF(bufA, bufB, true)
    }
    break;
  }
  asm volatile("s_waitcnt vmcnt(0)" ::: "memory");
  if (lane == 0)
    __hip_atomic_store((u32*)(p.ws[dest] + OFF_FE) + rank * 32 + j, seq, __ATOMIC_RELAXED, __HIP_MEMORY_SCOPE_SYSTEM);
  const u32* fe = (const u32*)(p.ws[rank] + OFF_FE) + dest * 32 + j;
  while (ok && __hip_atomic_load(fe, __ATOMIC_RELAXED, __HIP_MEMORY_SCOPE_SYSTEM) < seq)
    if (__builtin_amdgcn_s_memrealtime() - t0 > DEADLINE) ok = false;
  if (!ok && lane == 0) p.out[rank][oc] = 0x7f;
}

#define GL4(so) "buffer_load_dwordx4 %[voff], %[rs], " so " offen sc1 lds\n\t"                 \
                "buffer_load_dwordx4 %[voff], %[rs], " so " offen offset:1024 sc1 lds\n\t"     \
                "buffer_load_dwordx4 %[voff], %[rs], " so " offen offset:2048 sc1 lds\n\t"     \
                "buffer_load_dwordx4 %[voff], %[rs], " so " offen offset:3072 sc1 lds\n\t"
#define NEXT4(so) "s_add_u32 m0, m0, 0x1000\n\ts_add_u32 " so ", " so ", 0x1000\n\t"
__device__ __forceinline__ bool load_job(u32 lw, u32 isB, u32 U, u32 v, u32& n, char* myws, u32 lane, u32 seq, u64 t0) {
  const rsrc_t rs = mk_rsrc(myws, WS_USED);
  const u32* fx = (const u32*)(myws + OFF_FX) + (isB * 16 + 2 * U) * 32 + lane;
  const u32* fw = (const u32*)(myws + OFF_FW) + (2 * v) * 32 + lane;
  const u32* fp = (const u32*)(myws + OFF_FP + (U * 16 + v) * 64) + (lane & 3);
  const u32 xbase = (isB ? OFF_XO : OFF_XS) + 2 * U * 32 * PIECE, wbase = OFF_W + v * 64 * PIECE;
  u32 rem = 0x11111111u << lw, rdy = 0, needP = isB && lw == 0, pre = 32;
  while (rem | needP) {
    u32 cand = rdy & rem, part = 3;
    if (!cand) {
      const u32 a = __hip_atomic_load(fx, __ATOMIC_RELAXED, __HIP_MEMORY_SCOPE_SYSTEM);
      const u32 c = __hip_atomic_load(fw, __ATOMIC_RELAXED, __HIP_MEMORY_SCOPE_SYSTEM);
      if (needP && __all(__hip_atomic_load(fp, __ATOMIC_RELAXED, __HIP_MEMORY_SCOPE_SYSTEM) >= seq)) {
        asm volatile("ds_or_b32 %0, %1" ::"v"(C_CTL), "v"(lane == 0 ? 2u : 0u) : "memory");
        needP = 0;
      }
      const u64 m = __ballot(a >= seq && c >= seq), mw = __ballot(c >= seq);
      rdy = (u32)m & (u32)(m >> 32);
      if (__builtin_amdgcn_s_memrealtime() - t0 > DEADLINE) return false;
      if (!isB || pre != 32 || (rdy & rem)) continue;
      const u32 wr = (u32)mw & (u32)(mw >> 32) & rem;
#pragma unroll
      for (int ks = 4; ks >= 0; --ks)
        if (wr & (0x8421u << ks)) cand = wr & (0x8421u << ks);
      if (!cand) continue;
      part = 2;
    }
    u32 step = __builtin_ctz(cand);
    if (part == 2) {
      pre = step;
    } else {
      if (pre != 32 && ((cand >> pre) & 1)) { step = pre; part = 1; }
      pre = 32;
      rem &= ~(1u << step);
    }
    u32 x = xbase + step * PIECE, w = wbase + step * 2 * PIECE, tmp, spin = 0;
    asm volatile(
        "1:\n\t"
        "ds_read_b128 v[52:55], %[acons]\n\t"
        "s_waitcnt lgkmcnt(0)\n\t"
        "v_min_u32 v52, v52, v53\n\t"
        "v_min3_u32 v52, v52, v54, v55\n\t"
        "s_add_u32 %[spin], %[spin], 1\n\t"
        "v_readfirstlane_b32 %[tmp], v52\n\t"
        "s_cmp_eq_u32 %[tmp], %[n]\n\t"
        "s_cbranch_scc1 2f\n\t"
        "s_bitcmp0_b32 %[spin], 23\n\t"
        "s_cbranch_scc1 1b\n\t"
        "s_branch 3f\n\t"
        "2:\n\t"
        "s_mov_b32 %[spin], 0\n\t"
        "s_bitcmp0_b32 %[part], 0\n\t"
        "s_cbranch_scc1 4f\n\t"
        "s_mov_b32 m0, %[m0]\n\t"
        "s_nop 0\n\t"
        GL4("%[x]") NEXT4("%[x]") GL4("%[x]")
        "s_add_u32 m0, m0, 0x1000\n\ts_add_u32 %[x], %[x], 0x3f000\n\t"
        GL4("%[x]") NEXT4("%[x]") GL4("%[x]")
        "s_bitcmp0_b32 %[part], 1\n\t"
        "s_cbranch_scc1 5f\n\t"
        "4:\n\t"
        "s_add_u32 m0, %[m0], 0x4000\n\t"
        "s_nop 0\n\t"
        GL4("%[w]") NEXT4("%[w]") GL4("%[w]") NEXT4("%[w]") GL4("%[w]") NEXT4("%[w]") GL4("%[w]")
        "s_bitcmp0_b32 %[part], 0\n\t"
        "s_cbranch_scc1 3f\n\t"
        "5:\n\t"
        "s_add_u32 %[n], %[n], 1\n\t"
        "v_mov_b32 v52, %[n]\n\t"
        "s_waitcnt vmcnt(0)\n\t"
        "ds_write_b32 %[afull], v52\n\t"
        "3:"
        : [x] "+&s"(x), [w] "+&s"(w), [tmp] "=&s"(tmp), [spin] "+&s"(spin), [n] "+&s"(n)
        : [voff] "v"(lane * 16), [rs] "s"(rs), [acons] "v"(C_CONS + lw * 16), [afull] "v"(C_FULL + lw * 4), [m0] "s"(lw * SLOT), [part] "s"(part)
        : "scc", "memory", "v52", "v53", "v54", "v55");
    if (spin) return false;
  }
  return true;
}

#define TRY(k, n) "s_cmp_gt_u32 %[f" #k "], " n "\n\ts_cbranch_scc1 2" #k "f\n\t"
#define TAKE(k, n, base)                                         \
      "2" #k ":\n\t"                                             \
      "s_add_u32 " n ", " n ", 1\n\t"                            \
      "s_add_u32 %[xb], %[xoff], " base "\n\t"                   \
      "s_add_u32 %[wb], %[woff], " base "\n\t"                   \
      "v_add_u32 v50, " #k "*16, %[ac]\n\t"                      \
      "v_mov_b32 v51, " n "\n\t"
#define RDSTATE "ds_read_b128 v[52:55], v48\n\tds_read_b32 v49, v48 offset:16\n\t"

__global__ void __launch_bounds__(512, 2) gemm_ar_q8_fused(Params p) {
  extern __shared__ __attribute__((aligned(16))) char lds[];
  const u32 lane = threadIdx.x & 63, wid = rfl(threadIdx.x >> 6), b = blockIdx.x, rank = p.rank;
  if ((u32)(size_t)lds != 0) __builtin_trap();
  char* const myws = p.ws[rank];
  const u64 t0 = __builtin_amdgcn_s_memrealtime();
  const u32 seqv = __hip_atomic_load((u32*)(myws + OFF_CNT + b * 64), __ATOMIC_RELAXED, __HIP_MEMORY_SCOPE_AGENT);
  const u32 xg = b & 7, jb = b >> 3;
  if (wid == 0) STAMP(0);
  if (jb < NSJ) {
    u32 hi;
    asm volatile("s_getreg_b32 %0, hwreg(HW_REG_XCC_ID)" : "=s"(hi));
    hi = (hi >> 2) & 1;
    const u32 sh = jb * 4 + (xg & 3);
    if (wid < SW) {
      u32 dest, j;
      for (;; hi ^= 1) {
        const u32 nl = hi ? 4 : 3, d = sh % nl;
        const u32 glist = (((0xB2u >> rank) & 1) ^ hi) ? 0x7541u : 0x6320u;
        dest = (glist >> (4 * d)) & 7;
        if (!hi && dest >= rank) dest = (glist >> (4 * d + 4)) & 7;
        j = (sh / nl) * SW + wid;
        const u32 toff = OFF_R + ((sh * 2 + hi) * SW + wid) * 4;
        if (j < WR) {
          if (!send_stream<true>(p, dest, j, WR, OFF_CNT + b * 64, toff, lane, t0)) break;
        } else {
          u32 old = seqv;
          asm volatile("" : "+v"(old));
          const u32 s1 = rfl(old) + 1;
          if (lane == 0) old = __hip_atomic_exchange((u32*)(myws + toff), s1, __ATOMIC_RELAXED, __HIP_MEMORY_SCOPE_AGENT);
          if (rfl(old) != s1) break;
        }
      }
      u32 s0 = seqv;
      asm volatile("" : "+v"(s0));
      const u32 seq = rfl(s0) + 1;
      SSTAMP(b * 128 + (8 + wid) * 8);
      if (j < WR3) send_out(p, dest, j, seq, lane, t0);
      SSTAMP(256 * 128 + b * 128 + wid * 8);
    }
    u32 sv = seqv;
    asm volatile("" : "+v"(sv));
    const u32 seq = rfl(sv) + 1;
    __syncthreads();
    if (threadIdx.x == 0) __hip_atomic_store((u32*)(myws + OFF_CNT + b * 64), seq, __ATOMIC_RELAXED, __HIP_MEMORY_SCOPE_AGENT);
    if (b == 0 && threadIdx.x < 8)
      __hip_atomic_store((u64*)(myws + OFF_Q + ((~seq & 1) * 8 + threadIdx.x) * 64), 0ull, __ATOMIC_RELAXED, __HIP_MEMORY_SCOPE_AGENT);
    return;
  }
  const u32 isB1 = jb >= NSJ + 16, i1 = isB1 ? jb - NSJ - 16 : jb - NSJ, njobs = (!isB1 && i1 < NSJ) ? 2 : 1, i2 = 16 - NSJ + i1;
  const u32 U1 = (xg & 1) * 4 + (i1 & 3), o1 = (xg >> 1) + 4 * (i1 >> 2), v1 = (o1 >> 1) + 8 * (o1 & 1);
  const u32 U2 = (xg & 1) * 4 + (i2 & 3), o2 = (xg >> 1) + 4 * (i2 >> 2), v2 = (o2 >> 1) + 8 * (o2 & 1);
  const u32 isB = isB1 || njobs == 2, UB = isB1 ? U1 : U2, vB = isB1 ? v1 : v2, tileB = UB * 16 + vB;
  if (wid == 0) lds_st32(CTRL + lane * 4, 0);
  asm volatile("s_waitcnt lgkmcnt(0)\n\ts_barrier" ::: "memory");
  const u32 wm = wid & 1, wn = (wid >> 1) & 1;
  float y[64];
#pragma unroll
  for (int i = 0; i < 64; ++i) y[i] = 0.f;

  if (wid < 4) {
    asm volatile("v_accvgpr_write_b32 a0, 0\n\tv_accvgpr_write_b32 a1, 0\n\tv_accvgpr_write_b32 a2, 0\n\tv_accvgpr_write_b32 a3, 0\n\tv_accvgpr_write_b32 a4, 0\n\tv_accvgpr_write_b32 a5, 0\n\tv_accvgpr_write_b32 a6, 0\n\tv_accvgpr_write_b32 a7, 0\n\tv_accvgpr_write_b32 a8, 0\n\tv_accvgpr_write_b32 a9, 0\n\tv_accvgpr_write_b32 a10, 0\n\tv_accvgpr_write_b32 a11, 0\n\tv_accvgpr_write_b32 a12, 0\n\tv_accvgpr_write_b32 a13, 0\n\tv_accvgpr_write_b32 a14, 0\n\tv_accvgpr_write_b32 a15, 0\n\tv_accvgpr_write_b32 a16, 0\n\tv_accvgpr_write_b32 a17, 0\n\tv_accvgpr_write_b32 a18, 0\n\tv_accvgpr_write_b32 a19, 0\n\tv_accvgpr_write_b32 a20, 0\n\tv_accvgpr_write_b32 a21, 0\n\tv_accvgpr_write_b32 a22, 0\n\tv_accvgpr_write_b32 a23, 0\n\tv_accvgpr_write_b32 a24, 0\n\tv_accvgpr_write_b32 a25, 0\n\tv_accvgpr_write_b32 a26, 0\n\tv_accvgpr_write_b32 a27, 0\n\tv_accvgpr_write_b32 a28, 0\n\tv_accvgpr_write_b32 a29, 0\n\tv_accvgpr_write_b32 a30, 0\n\tv_accvgpr_write_b32 a31, 0\n\tv_accvgpr_write_b32 a32, 0\n\tv_accvgpr_write_b32 a33, 0\n\tv_accvgpr_write_b32 a34, 0\n\tv_accvgpr_write_b32 a35, 0\n\tv_accvgpr_write_b32 a36, 0\n\tv_accvgpr_write_b32 a37, 0\n\tv_accvgpr_write_b32 a38, 0\n\tv_accvgpr_write_b32 a39, 0\n\tv_accvgpr_write_b32 a40, 0\n\tv_accvgpr_write_b32 a41, 0\n\tv_accvgpr_write_b32 a42, 0\n\tv_accvgpr_write_b32 a43, 0\n\tv_accvgpr_write_b32 a44, 0\n\tv_accvgpr_write_b32 a45, 0\n\tv_accvgpr_write_b32 a46, 0\n\tv_accvgpr_write_b32 a47, 0\n\tv_accvgpr_write_b32 a48, 0\n\tv_accvgpr_write_b32 a49, 0\n\tv_accvgpr_write_b32 a50, 0\n\tv_accvgpr_write_b32 a51, 0\n\tv_accvgpr_write_b32 a52, 0\n\tv_accvgpr_write_b32 a53, 0\n\tv_accvgpr_write_b32 a54, 0\n\tv_accvgpr_write_b32 a55, 0\n\tv_accvgpr_write_b32 a56, 0\n\tv_accvgpr_write_b32 a57, 0\n\tv_accvgpr_write_b32 a58, 0\n\tv_accvgpr_write_b32 a59, 0\n\tv_accvgpr_write_b32 a60, 0\n\tv_accvgpr_write_b32 a61, 0\n\tv_accvgpr_write_b32 a62, 0\n\tv_accvgpr_write_b32 a63, 0" ::: "a0", "a1", "a2", "a3", "a4", "a5", "a6", "a7", "a8", "a9", "a10", "a11", "a12", "a13", "a14", "a15", "a16", "a17", "a18", "a19", "a20", "a21", "a22", "a23", "a24", "a25", "a26", "a27", "a28", "a29", "a30", "a31", "a32", "a33", "a34", "a35", "a36", "a37", "a38", "a39", "a40", "a41", "a42", "a43", "a44", "a45", "a46", "a47", "a48", "a49", "a50", "a51", "a52", "a53", "a54", "a55", "a56", "a57", "a58", "a59", "a60", "a61", "a62", "a63");
    u32 fo[4];
    {
      const u32 i = lane & 31, h = lane >> 5;
#pragma unroll
      for (int s = 0; s < 4; ++s) fo[s] = i * 128 + (((2 * s + h) ^ ((i >> 1) & 7)) << 4);
    }
    const rsrc_t rs = mk_rsrc(myws, WS_USED);
    const u32 seq = rfl(seqv) + 1;
    u32 n0 = 0, n1 = 0, n2 = 0, n3 = 0, done = 0;
    const u32 total = 32 * njobs;
    u32 want = isB ? 3 : 1, stop = 32, inA = !isB1;
    for (;;) {
      u32 f0, f1, f2, f3, ctl, xb, wb;
      asm volatile(
      "v_mov_b32 v48, 0x20000\n\t"
      RDSTATE
      "1:\n\t"
      "s_waitcnt lgkmcnt(0)\n\t"
      "v_readfirstlane_b32 %[ctl], v49\n\t"
      "v_readfirstlane_b32 %[f0], v52\n\t"
      "v_readfirstlane_b32 %[f1], v53\n\t"
      "v_readfirstlane_b32 %[f2], v54\n\t"
      "v_readfirstlane_b32 %[f3], v55\n\t"
      "s_and_b32 %[xb], %[ctl], %[want]\n\t"
      "s_cbranch_scc1 9f\n\t"
      TRY(0, "%[n0]") TRY(1, "%[n1]") TRY(2, "%[n2]") TRY(3, "%[n3]")
      "s_sleep 1\n\t"
      RDSTATE
      "s_branch 1b\n\t"
      TAKE(0, "%[n0]", "0") "s_branch 6f\n\t"
      TAKE(1, "%[n1]", "0x8000") "s_branch 6f\n\t"
      TAKE(2, "%[n2]", "0x10000") "s_branch 6f\n\t"
      TAKE(3, "%[n3]", "0x18000")
      "6:\n\t"
      "v_add_u32 v56, %[xb], %[fo0]\n\tv_add_u32 v60, %[wb], %[fo0]\n\t"
      "v_add_u32 v57, %[xb], %[fo1]\n\tv_add_u32 v61, %[wb], %[fo1]\n\t"
      "v_add_u32 v58, %[xb], %[fo2]\n\tv_add_u32 v62, %[wb], %[fo2]\n\t"
      "v_add_u32 v59, %[xb], %[fo3]\n\tv_add_u32 v63, %[wb], %[fo3]\n\t"
      "ds_read_b128 v[64:67], v56\n\tds_read_b128 v[72:75], v60\n\tds_read_b128 v[68:71], v56 offset:4096\n\tds_read_b128 v[76:79], v60 offset:4096\n\t"
      "ds_read_b128 v[80:83], v57\n\tds_read_b128 v[88:91], v61\n\tds_read_b128 v[84:87], v57 offset:4096\n\tds_read_b128 v[92:95], v61 offset:4096\n\t"
      "ds_read_b128 v[96:99], v58\n\tds_read_b128 v[104:107], v62\n\tds_read_b128 v[100:103], v58 offset:4096\n\tds_read_b128 v[108:111], v62 offset:4096\n\t"
      "s_waitcnt lgkmcnt(8)\n\t"
      "v_mfma_f32_32x32x16_bf16 a[0:15], v[72:75], v[64:67], a[0:15]\n\t"
      "v_mfma_f32_32x32x16_bf16 a[16:31], v[76:79], v[64:67], a[16:31]\n\t"
      "v_mfma_f32_32x32x16_bf16 a[32:47], v[72:75], v[68:71], a[32:47]\n\t"
      "v_mfma_f32_32x32x16_bf16 a[48:63], v[76:79], v[68:71], a[48:63]\n\t"
      "ds_read_b128 v[112:115], v59\n\tds_read_b128 v[120:123], v63\n\tds_read_b128 v[116:119], v59 offset:4096\n\tds_read_b128 v[124:127], v63 offset:4096\n\t"
      "ds_write_b32 v50, v51\n\t"
      RDSTATE
      "s_waitcnt lgkmcnt(11)\n\t"
      "v_mfma_f32_32x32x16_bf16 a[0:15], v[88:91], v[80:83], a[0:15]\n\t"
      "v_mfma_f32_32x32x16_bf16 a[16:31], v[92:95], v[80:83], a[16:31]\n\t"
      "v_mfma_f32_32x32x16_bf16 a[32:47], v[88:91], v[84:87], a[32:47]\n\t"
      "v_mfma_f32_32x32x16_bf16 a[48:63], v[92:95], v[84:87], a[48:63]\n\t"
      "s_waitcnt lgkmcnt(7)\n\t"
      "v_mfma_f32_32x32x16_bf16 a[0:15], v[104:107], v[96:99], a[0:15]\n\t"
      "v_mfma_f32_32x32x16_bf16 a[16:31], v[108:111], v[96:99], a[16:31]\n\t"
      "v_mfma_f32_32x32x16_bf16 a[32:47], v[104:107], v[100:103], a[32:47]\n\t"
      "v_mfma_f32_32x32x16_bf16 a[48:63], v[108:111], v[100:103], a[48:63]\n\t"
      "s_waitcnt lgkmcnt(3)\n\t"
      "v_mfma_f32_32x32x16_bf16 a[0:15], v[120:123], v[112:115], a[0:15]\n\t"
      "v_mfma_f32_32x32x16_bf16 a[16:31], v[124:127], v[112:115], a[16:31]\n\t"
      "v_mfma_f32_32x32x16_bf16 a[32:47], v[120:123], v[116:119], a[32:47]\n\t"
      "v_mfma_f32_32x32x16_bf16 a[48:63], v[124:127], v[116:119], a[48:63]\n\t"
      "s_add_u32 %[done], %[done], 1\n\t"
      "s_cmp_lt_u32 %[done], %[stop]\n\t"
      "s_cbranch_scc1 1b\n\t"
      "9:\n\t"
      "s_waitcnt lgkmcnt(0)\n\t"
      "s_nop 15"
      : [n0] "+&s"(n0), [n1] "+&s"(n1), [n2] "+&s"(n2), [n3] "+&s"(n3), [done] "+&s"(done),
        [f0] "=&s"(f0), [f1] "=&s"(f1), [f2] "=&s"(f2), [f3] "=&s"(f3), [ctl] "=&s"(ctl), [xb] "=&s"(xb), [wb] "=&s"(wb)
      : [fo0] "v"(fo[0]), [fo1] "v"(fo[1]), [fo2] "v"(fo[2]), [fo3] "v"(fo[3]), [ac] "v"(C_CONS + wid * 4),
        [want] "s"(want), [stop] "s"(stop), [xoff] "s"(wm * PIECE), [woff] "s"(2 * PIECE + wn * PIECE)
      : "scc", "memory", "v48", "v49", "v50", "v51", "v52", "v53", "v54", "v55", "v56", "v57", "v58", "v59", "v60", "v61", "v62", "v63", "v64", "v65", "v66", "v67", "v68", "v69", "v70", "v71", "v72", "v73", "v74", "v75", "v76", "v77", "v78", "v79", "v80", "v81", "v82", "v83", "v84", "v85", "v86", "v87", "v88", "v89", "v90", "v91", "v92", "v93", "v94", "v95", "v96", "v97", "v98", "v99", "v100", "v101", "v102", "v103", "v104", "v105", "v106", "v107", "v108", "v109", "v110", "v111", "v112", "v113", "v114", "v115", "v116", "v117", "v118", "v119", "v120", "v121", "v122", "v123", "v124", "v125", "v126", "v127", "a0", "a1", "a2", "a3", "a4", "a5", "a6", "a7", "a8", "a9", "a10", "a11", "a12", "a13", "a14", "a15", "a16", "a17", "a18", "a19", "a20", "a21", "a22", "a23", "a24", "a25", "a26", "a27", "a28", "a29", "a30", "a31", "a32", "a33", "a34", "a35", "a36", "a37", "a38", "a39", "a40", "a41", "a42", "a43", "a44", "a45", "a46", "a47", "a48", "a49", "a50", "a51", "a52", "a53", "a54", "a55", "a56", "a57", "a58", "a59", "a60", "a61", "a62", "a63");
      if (ctl & 1) break;
      if (inA && done == 32) {
        const u32 tileA = U1 * 16 + v1;
        inA = 0;
#define PACK_STORE(J0)                                                                              \
        {                                                                                           \
          _Pragma("unroll") for (int j = 0; j < 2; ++j) {                                           \
            u32x4 o4;                                                                               \
            _Pragma("unroll") for (int e = 0; e < 4; ++e) {                                         \
              f32x2 f = {d[8 * j + 2 * e], d[8 * j + 2 * e + 1]};                                   \
              o4[e] = __builtin_bit_cast(u32, __builtin_convertvector(f, bf16x2));                  \
            }                                                                                       \
            st128(o4, rs, lane * 16, OFF_PO + tileA * 32768 + wid * PIECE + (J0 + j) * 1024);       \
          }                                                                                         \
        }
        { float d[16]; asm volatile("v_accvgpr_read_b32 %0, a0\n\tv_accvgpr_read_b32 %1, a1\n\tv_accvgpr_read_b32 %2, a2\n\tv_accvgpr_read_b32 %3, a3\n\tv_accvgpr_read_b32 %4, a4\n\tv_accvgpr_read_b32 %5, a5\n\tv_accvgpr_read_b32 %6, a6\n\tv_accvgpr_read_b32 %7, a7\n\tv_accvgpr_read_b32 %8, a8\n\tv_accvgpr_read_b32 %9, a9\n\tv_accvgpr_read_b32 %10, a10\n\tv_accvgpr_read_b32 %11, a11\n\tv_accvgpr_read_b32 %12, a12\n\tv_accvgpr_read_b32 %13, a13\n\tv_accvgpr_read_b32 %14, a14\n\tv_accvgpr_read_b32 %15, a15" : "=v"(d[0]), "=v"(d[1]), "=v"(d[2]), "=v"(d[3]), "=v"(d[4]), "=v"(d[5]), "=v"(d[6]), "=v"(d[7]), "=v"(d[8]), "=v"(d[9]), "=v"(d[10]), "=v"(d[11]), "=v"(d[12]), "=v"(d[13]), "=v"(d[14]), "=v"(d[15])); PACK_STORE(0) }
        { float d[16]; asm volatile("v_accvgpr_read_b32 %0, a16\n\tv_accvgpr_read_b32 %1, a17\n\tv_accvgpr_read_b32 %2, a18\n\tv_accvgpr_read_b32 %3, a19\n\tv_accvgpr_read_b32 %4, a20\n\tv_accvgpr_read_b32 %5, a21\n\tv_accvgpr_read_b32 %6, a22\n\tv_accvgpr_read_b32 %7, a23\n\tv_accvgpr_read_b32 %8, a24\n\tv_accvgpr_read_b32 %9, a25\n\tv_accvgpr_read_b32 %10, a26\n\tv_accvgpr_read_b32 %11, a27\n\tv_accvgpr_read_b32 %12, a28\n\tv_accvgpr_read_b32 %13, a29\n\tv_accvgpr_read_b32 %14, a30\n\tv_accvgpr_read_b32 %15, a31" : "=v"(d[0]), "=v"(d[1]), "=v"(d[2]), "=v"(d[3]), "=v"(d[4]), "=v"(d[5]), "=v"(d[6]), "=v"(d[7]), "=v"(d[8]), "=v"(d[9]), "=v"(d[10]), "=v"(d[11]), "=v"(d[12]), "=v"(d[13]), "=v"(d[14]), "=v"(d[15])); PACK_STORE(2) }
        { float d[16]; asm volatile("v_accvgpr_read_b32 %0, a32\n\tv_accvgpr_read_b32 %1, a33\n\tv_accvgpr_read_b32 %2, a34\n\tv_accvgpr_read_b32 %3, a35\n\tv_accvgpr_read_b32 %4, a36\n\tv_accvgpr_read_b32 %5, a37\n\tv_accvgpr_read_b32 %6, a38\n\tv_accvgpr_read_b32 %7, a39\n\tv_accvgpr_read_b32 %8, a40\n\tv_accvgpr_read_b32 %9, a41\n\tv_accvgpr_read_b32 %10, a42\n\tv_accvgpr_read_b32 %11, a43\n\tv_accvgpr_read_b32 %12, a44\n\tv_accvgpr_read_b32 %13, a45\n\tv_accvgpr_read_b32 %14, a46\n\tv_accvgpr_read_b32 %15, a47" : "=v"(d[0]), "=v"(d[1]), "=v"(d[2]), "=v"(d[3]), "=v"(d[4]), "=v"(d[5]), "=v"(d[6]), "=v"(d[7]), "=v"(d[8]), "=v"(d[9]), "=v"(d[10]), "=v"(d[11]), "=v"(d[12]), "=v"(d[13]), "=v"(d[14]), "=v"(d[15])); PACK_STORE(4) }
        { float d[16]; asm volatile("v_accvgpr_read_b32 %0, a48\n\tv_accvgpr_read_b32 %1, a49\n\tv_accvgpr_read_b32 %2, a50\n\tv_accvgpr_read_b32 %3, a51\n\tv_accvgpr_read_b32 %4, a52\n\tv_accvgpr_read_b32 %5, a53\n\tv_accvgpr_read_b32 %6, a54\n\tv_accvgpr_read_b32 %7, a55\n\tv_accvgpr_read_b32 %8, a56\n\tv_accvgpr_read_b32 %9, a57\n\tv_accvgpr_read_b32 %10, a58\n\tv_accvgpr_read_b32 %11, a59\n\tv_accvgpr_read_b32 %12, a60\n\tv_accvgpr_read_b32 %13, a61\n\tv_accvgpr_read_b32 %14, a62\n\tv_accvgpr_read_b32 %15, a63" : "=v"(d[0]), "=v"(d[1]), "=v"(d[2]), "=v"(d[3]), "=v"(d[4]), "=v"(d[5]), "=v"(d[6]), "=v"(d[7]), "=v"(d[8]), "=v"(d[9]), "=v"(d[10]), "=v"(d[11]), "=v"(d[12]), "=v"(d[13]), "=v"(d[14]), "=v"(d[15])); PACK_STORE(6) }
        asm volatile("v_accvgpr_write_b32 a0, 0\n\tv_accvgpr_write_b32 a1, 0\n\tv_accvgpr_write_b32 a2, 0\n\tv_accvgpr_write_b32 a3, 0\n\tv_accvgpr_write_b32 a4, 0\n\tv_accvgpr_write_b32 a5, 0\n\tv_accvgpr_write_b32 a6, 0\n\tv_accvgpr_write_b32 a7, 0\n\tv_accvgpr_write_b32 a8, 0\n\tv_accvgpr_write_b32 a9, 0\n\tv_accvgpr_write_b32 a10, 0\n\tv_accvgpr_write_b32 a11, 0\n\tv_accvgpr_write_b32 a12, 0\n\tv_accvgpr_write_b32 a13, 0\n\tv_accvgpr_write_b32 a14, 0\n\tv_accvgpr_write_b32 a15, 0\n\tv_accvgpr_write_b32 a16, 0\n\tv_accvgpr_write_b32 a17, 0\n\tv_accvgpr_write_b32 a18, 0\n\tv_accvgpr_write_b32 a19, 0\n\tv_accvgpr_write_b32 a20, 0\n\tv_accvgpr_write_b32 a21, 0\n\tv_accvgpr_write_b32 a22, 0\n\tv_accvgpr_write_b32 a23, 0\n\tv_accvgpr_write_b32 a24, 0\n\tv_accvgpr_write_b32 a25, 0\n\tv_accvgpr_write_b32 a26, 0\n\tv_accvgpr_write_b32 a27, 0\n\tv_accvgpr_write_b32 a28, 0\n\tv_accvgpr_write_b32 a29, 0\n\tv_accvgpr_write_b32 a30, 0\n\tv_accvgpr_write_b32 a31, 0\n\tv_accvgpr_write_b32 a32, 0\n\tv_accvgpr_write_b32 a33, 0\n\tv_accvgpr_write_b32 a34, 0\n\tv_accvgpr_write_b32 a35, 0\n\tv_accvgpr_write_b32 a36, 0\n\tv_accvgpr_write_b32 a37, 0\n\tv_accvgpr_write_b32 a38, 0\n\tv_accvgpr_write_b32 a39, 0\n\tv_accvgpr_write_b32 a40, 0\n\tv_accvgpr_write_b32 a41, 0\n\tv_accvgpr_write_b32 a42, 0\n\tv_accvgpr_write_b32 a43, 0\n\tv_accvgpr_write_b32 a44, 0\n\tv_accvgpr_write_b32 a45, 0\n\tv_accvgpr_write_b32 a46, 0\n\tv_accvgpr_write_b32 a47, 0\n\tv_accvgpr_write_b32 a48, 0\n\tv_accvgpr_write_b32 a49, 0\n\tv_accvgpr_write_b32 a50, 0\n\tv_accvgpr_write_b32 a51, 0\n\tv_accvgpr_write_b32 a52, 0\n\tv_accvgpr_write_b32 a53, 0\n\tv_accvgpr_write_b32 a54, 0\n\tv_accvgpr_write_b32 a55, 0\n\tv_accvgpr_write_b32 a56, 0\n\tv_accvgpr_write_b32 a57, 0\n\tv_accvgpr_write_b32 a58, 0\n\tv_accvgpr_write_b32 a59, 0\n\tv_accvgpr_write_b32 a60, 0\n\tv_accvgpr_write_b32 a61, 0\n\tv_accvgpr_write_b32 a62, 0\n\tv_accvgpr_write_b32 a63, 0" ::: "a0", "a1", "a2", "a3", "a4", "a5", "a6", "a7", "a8", "a9", "a10", "a11", "a12", "a13", "a14", "a15", "a16", "a17", "a18", "a19", "a20", "a21", "a22", "a23", "a24", "a25", "a26", "a27", "a28", "a29", "a30", "a31", "a32", "a33", "a34", "a35", "a36", "a37", "a38", "a39", "a40", "a41", "a42", "a43", "a44", "a45", "a46", "a47", "a48", "a49", "a50", "a51", "a52", "a53", "a54", "a55", "a56", "a57", "a58", "a59", "a60", "a61", "a62", "a63");
        asm volatile("s_waitcnt vmcnt(0)" ::: "memory");
        if (lane == 0) __builtin_amdgcn_raw_buffer_store_b32(seq, rs, 0, OFF_FP + tileA * 64 + 32 + wid * 4, AUX_SYS);
        stop = total;
        if (wid == 0) STAMP(2);
      }
#define PLOAD                                                                                       \
        {                                                                                           \
          const u32 so = OFF_P + tileB * 32768 + wid * PIECE;                                       \
          const u32 xe = (lane >> 3) * 128 + (((lane & 7) ^ (lane >> 4)) << 4);                     \
          asm volatile("buffer_load_dwordx4 a[64:67], %0, %2, %3 offen offset:0 sc0 sc1\n\tbuffer_load_dwordx4 a[68:71], %1, %2, %3 offen offset:1024 sc0 sc1\n\tbuffer_load_dwordx4 a[72:75], %0, %2, %3 offen offset:2048 sc0 sc1\n\tbuffer_load_dwordx4 a[76:79], %1, %2, %3 offen offset:3072 sc0 sc1\n\tbuffer_load_dwordx4 a[80:83], %0, %2, %4 offen offset:0 sc0 sc1\n\tbuffer_load_dwordx4 a[84:87], %1, %2, %4 offen offset:1024 sc0 sc1\n\tbuffer_load_dwordx4 a[88:91], %0, %2, %4 offen offset:2048 sc0 sc1\n\tbuffer_load_dwordx4 a[92:95], %1, %2, %4 offen offset:3072 sc0 sc1" :: "v"(xe), "v"(xe ^ 64), "s"(rs), "s"(so), "s"(so + 4096) : "memory", "a64", "a65", "a66", "a67", "a68", "a69", "a70", "a71", "a72", "a73", "a74", "a75", "a76", "a77", "a78", "a79", "a80", "a81", "a82", "a83", "a84", "a85", "a86", "a87", "a88", "a89", "a90", "a91", "a92", "a93", "a94", "a95"); \
        }
      if (ctl & want & 2) {
        PLOAD
        want = 1;
      }
      if (done == total) {
        if (want == 1) break;
        const u32* fp = (const u32*)(myws + OFF_FP + tileB * 64) + wid;
        while (__hip_atomic_load(fp, __ATOMIC_RELAXED, __HIP_MEMORY_SCOPE_SYSTEM) < seq)
          if (__builtin_amdgcn_s_memrealtime() - t0 > DEADLINE) { lds_st32(C_CTL, 1); break; }
        PLOAD
        break;
      }
    }
    if (isB) {
      asm volatile("s_waitcnt vmcnt(0)" ::: "memory");
      { float* d = y; asm volatile("v_accvgpr_read_b32 %0, a0\n\tv_accvgpr_read_b32 %1, a1\n\tv_accvgpr_read_b32 %2, a2\n\tv_accvgpr_read_b32 %3, a3\n\tv_accvgpr_read_b32 %4, a4\n\tv_accvgpr_read_b32 %5, a5\n\tv_accvgpr_read_b32 %6, a6\n\tv_accvgpr_read_b32 %7, a7\n\tv_accvgpr_read_b32 %8, a8\n\tv_accvgpr_read_b32 %9, a9\n\tv_accvgpr_read_b32 %10, a10\n\tv_accvgpr_read_b32 %11, a11\n\tv_accvgpr_read_b32 %12, a12\n\tv_accvgpr_read_b32 %13, a13\n\tv_accvgpr_read_b32 %14, a14\n\tv_accvgpr_read_b32 %15, a15" : "=v"(d[0]), "=v"(d[1]), "=v"(d[2]), "=v"(d[3]), "=v"(d[4]), "=v"(d[5]), "=v"(d[6]), "=v"(d[7]), "=v"(d[8]), "=v"(d[9]), "=v"(d[10]), "=v"(d[11]), "=v"(d[12]), "=v"(d[13]), "=v"(d[14]), "=v"(d[15])); }
      { float* d = y + 16; asm volatile("v_accvgpr_read_b32 %0, a16\n\tv_accvgpr_read_b32 %1, a17\n\tv_accvgpr_read_b32 %2, a18\n\tv_accvgpr_read_b32 %3, a19\n\tv_accvgpr_read_b32 %4, a20\n\tv_accvgpr_read_b32 %5, a21\n\tv_accvgpr_read_b32 %6, a22\n\tv_accvgpr_read_b32 %7, a23\n\tv_accvgpr_read_b32 %8, a24\n\tv_accvgpr_read_b32 %9, a25\n\tv_accvgpr_read_b32 %10, a26\n\tv_accvgpr_read_b32 %11, a27\n\tv_accvgpr_read_b32 %12, a28\n\tv_accvgpr_read_b32 %13, a29\n\tv_accvgpr_read_b32 %14, a30\n\tv_accvgpr_read_b32 %15, a31" : "=v"(d[0]), "=v"(d[1]), "=v"(d[2]), "=v"(d[3]), "=v"(d[4]), "=v"(d[5]), "=v"(d[6]), "=v"(d[7]), "=v"(d[8]), "=v"(d[9]), "=v"(d[10]), "=v"(d[11]), "=v"(d[12]), "=v"(d[13]), "=v"(d[14]), "=v"(d[15])); }
      { float* d = y + 32; asm volatile("v_accvgpr_read_b32 %0, a32\n\tv_accvgpr_read_b32 %1, a33\n\tv_accvgpr_read_b32 %2, a34\n\tv_accvgpr_read_b32 %3, a35\n\tv_accvgpr_read_b32 %4, a36\n\tv_accvgpr_read_b32 %5, a37\n\tv_accvgpr_read_b32 %6, a38\n\tv_accvgpr_read_b32 %7, a39\n\tv_accvgpr_read_b32 %8, a40\n\tv_accvgpr_read_b32 %9, a41\n\tv_accvgpr_read_b32 %10, a42\n\tv_accvgpr_read_b32 %11, a43\n\tv_accvgpr_read_b32 %12, a44\n\tv_accvgpr_read_b32 %13, a45\n\tv_accvgpr_read_b32 %14, a46\n\tv_accvgpr_read_b32 %15, a47" : "=v"(d[0]), "=v"(d[1]), "=v"(d[2]), "=v"(d[3]), "=v"(d[4]), "=v"(d[5]), "=v"(d[6]), "=v"(d[7]), "=v"(d[8]), "=v"(d[9]), "=v"(d[10]), "=v"(d[11]), "=v"(d[12]), "=v"(d[13]), "=v"(d[14]), "=v"(d[15])); }
      { float* d = y + 48; asm volatile("v_accvgpr_read_b32 %0, a48\n\tv_accvgpr_read_b32 %1, a49\n\tv_accvgpr_read_b32 %2, a50\n\tv_accvgpr_read_b32 %3, a51\n\tv_accvgpr_read_b32 %4, a52\n\tv_accvgpr_read_b32 %5, a53\n\tv_accvgpr_read_b32 %6, a54\n\tv_accvgpr_read_b32 %7, a55\n\tv_accvgpr_read_b32 %8, a56\n\tv_accvgpr_read_b32 %9, a57\n\tv_accvgpr_read_b32 %10, a58\n\tv_accvgpr_read_b32 %11, a59\n\tv_accvgpr_read_b32 %12, a60\n\tv_accvgpr_read_b32 %13, a61\n\tv_accvgpr_read_b32 %14, a62\n\tv_accvgpr_read_b32 %15, a63" : "=v"(d[0]), "=v"(d[1]), "=v"(d[2]), "=v"(d[3]), "=v"(d[4]), "=v"(d[5]), "=v"(d[6]), "=v"(d[7]), "=v"(d[8]), "=v"(d[9]), "=v"(d[10]), "=v"(d[11]), "=v"(d[12]), "=v"(d[13]), "=v"(d[14]), "=v"(d[15])); }
      float mx = 0.f, nf = 0.f;
#define ADD_P(O)                                                                                    \
      _Pragma("unroll") for (int i = 0; i < 16; i += 2) {                                           \
        f32x2 a = {y[O + 2 * i], y[O + 2 * i + 1]}, c = {y[O + 2 * i + 2], y[O + 2 * i + 3]};       \
        a += (f32x2){__uint_as_float(d[i] << 16), __uint_as_float(d[i] & 0xffff0000u)};             \
        c += (f32x2){__uint_as_float(d[i + 1] << 16), __uint_as_float(d[i + 1] & 0xffff0000u)};     \
        y[O + 2 * i] = a[0]; y[O + 2 * i + 1] = a[1]; y[O + 2 * i + 2] = c[0]; y[O + 2 * i + 3] = c[1]; \
        mx = fmaxf(fmaxf(mx, fabsf(a[0])), fabsf(a[1]));                                            \
        mx = fmaxf(fmaxf(mx, fabsf(c[0])), fabsf(c[1]));                                            \
        sm = __builtin_elementwise_fma(a, (f32x2){0.f, 0.f}, __builtin_elementwise_fma(c, (f32x2){0.f, 0.f}, sm)); \
      }
      f32x2 sm = {0.f, 0.f};
      { u32 d[16]; asm volatile("v_accvgpr_read_b32 %0, a64\n\tv_accvgpr_read_b32 %1, a65\n\tv_accvgpr_read_b32 %2, a66\n\tv_accvgpr_read_b32 %3, a67\n\tv_accvgpr_read_b32 %4, a68\n\tv_accvgpr_read_b32 %5, a69\n\tv_accvgpr_read_b32 %6, a70\n\tv_accvgpr_read_b32 %7, a71\n\tv_accvgpr_read_b32 %8, a72\n\tv_accvgpr_read_b32 %9, a73\n\tv_accvgpr_read_b32 %10, a74\n\tv_accvgpr_read_b32 %11, a75\n\tv_accvgpr_read_b32 %12, a76\n\tv_accvgpr_read_b32 %13, a77\n\tv_accvgpr_read_b32 %14, a78\n\tv_accvgpr_read_b32 %15, a79" : "=v"(d[0]), "=v"(d[1]), "=v"(d[2]), "=v"(d[3]), "=v"(d[4]), "=v"(d[5]), "=v"(d[6]), "=v"(d[7]), "=v"(d[8]), "=v"(d[9]), "=v"(d[10]), "=v"(d[11]), "=v"(d[12]), "=v"(d[13]), "=v"(d[14]), "=v"(d[15])); ADD_P(0) }
      { u32 d[16]; asm volatile("v_accvgpr_read_b32 %0, a80\n\tv_accvgpr_read_b32 %1, a81\n\tv_accvgpr_read_b32 %2, a82\n\tv_accvgpr_read_b32 %3, a83\n\tv_accvgpr_read_b32 %4, a84\n\tv_accvgpr_read_b32 %5, a85\n\tv_accvgpr_read_b32 %6, a86\n\tv_accvgpr_read_b32 %7, a87\n\tv_accvgpr_read_b32 %8, a88\n\tv_accvgpr_read_b32 %9, a89\n\tv_accvgpr_read_b32 %10, a90\n\tv_accvgpr_read_b32 %11, a91\n\tv_accvgpr_read_b32 %12, a92\n\tv_accvgpr_read_b32 %13, a93\n\tv_accvgpr_read_b32 %14, a94\n\tv_accvgpr_read_b32 %15, a95" : "=v"(d[0]), "=v"(d[1]), "=v"(d[2]), "=v"(d[3]), "=v"(d[4]), "=v"(d[5]), "=v"(d[6]), "=v"(d[7]), "=v"(d[8]), "=v"(d[9]), "=v"(d[10]), "=v"(d[11]), "=v"(d[12]), "=v"(d[13]), "=v"(d[14]), "=v"(d[15])); ADD_P(32) }
      nf = sm[0] + sm[1];
      lds_st32(C_WMAX + wid * 4, wave_max(nf != nf ? 0x7fc00000u : __float_as_uint(mx)));
      if (wid == 0) STAMP(3);
    }
  } else {
    const u32 lw = wid - 4;
    send_stream<false>(p, rank, (b - 8 * NSJ) + (256 - 8 * NSJ) * lw, 4 * (256 - 8 * NSJ), OFF_CNT + b * 64, 0, lane, t0);
    if (lw == 0) STAMP(7);
    const u32 seq = rfl(seqv) + 1;
    u32 n = 0;
    {
      bool ok = load_job(lw, isB1, U1, v1, n, myws, lane, seq, t0);
      if (njobs == 2) {
        for (u32 fin = !ok; !fin;) {
          u32 c;
          asm volatile("ds_read_b32 %0, %1\n\ts_waitcnt lgkmcnt(0)" : "=v"(c) : "v"(C_CONS + (lane & 15) * 4) : "memory");
          fin = __all(c >= 8);
          if (__builtin_amdgcn_s_memrealtime() - t0 > DEADLINE) { ok = false; fin = 1; }
        }
        if (ok) ok = load_job(lw, 1, U2, v2, n, myws, lane, seq, t0);
      }
      if (!ok) asm volatile("ds_or_b32 %0, %1" ::"v"(C_CTL), "v"(1u) : "memory");
    }
  }
  asm volatile("s_waitcnt lgkmcnt(0)" ::: "memory");
  __syncthreads();
  const u32 seq = rfl(seqv) + 1;
  if (threadIdx.x == 0) __hip_atomic_store((u32*)(myws + OFF_CNT + b * 64), seq, __ATOMIC_RELAXED, __HIP_MEMORY_SCOPE_AGENT);
  if (!isB) return;
  {
    const u32x4 wmx = *(const u32x4*)(lds + C_WMAX);
    const u32 m = max(max(wmx[0], wmx[1]), max(wmx[2], wmx[3]));
    if (lane == 0)
      __hip_atomic_store((u64*)(p.ws[wid] + OFF_G) + rank * 128 + tileB, ((u64)seq << 32) | m, __ATOMIC_RELAXED, __HIP_MEMORY_SCOPE_SYSTEM);
  }
  const u32 r8 = lane >> 3, c = lane & 7;
  if (wid < 4) {
#pragma unroll
    for (int t = 0; t < 4; ++t) {
      const u32 m = wm * 64 + (t >> 1) * 32 + (lane & 31), ch = 16 * wn + 8 * (t & 1) + 4 * (lane >> 5);
#pragma unroll
      for (int e = 0; e < 4; ++e)
        *(f32x4*)(lds + m * 512 + (((ch + e) ^ (m & 7)) << 4)) = (f32x4){y[16 * t + 4 * e], y[16 * t + 4 * e + 1], y[16 * t + 4 * e + 2], y[16 * t + 4 * e + 3]};
    }
  }
  __syncthreads();
  f32x4 z[2][4];
#pragma unroll
  for (int t = 0; t < 2; ++t)
#pragma unroll
    for (int e = 0; e < 4; ++e) z[t][e] = *(const f32x4*)(lds + (16 * wid + 8 * t + r8) * 512 + (((4 * c + e) ^ r8) << 4));
  if (wid >= 4) {
    const rsrc_t rs = mk_rsrc(myws, WS_USED);
    u32 gm;
    for (;;) {
      const u32x4 g0 = __builtin_amdgcn_raw_buffer_load_b128(rs, lane * 16, OFF_G + (wid - 4) * 2048, AUX_SYS);
      const u32x4 g1 = __builtin_amdgcn_raw_buffer_load_b128(rs, lane * 16, OFF_G + (wid - 4) * 2048 + 1024, AUX_SYS);
      gm = max(max(g0[0], g0[2]), max(g1[0], g1[2]));
      if (__all(g0[1] == seq && g0[3] == seq && g1[1] == seq && g1[3] == seq)) break;
      if (__builtin_amdgcn_s_memrealtime() - t0 > DEADLINE) { lds_st32(C_CTL, 1); break; }
      asm volatile("" ::: "memory");
    }
    lds_st32(C_GMAX + (wid - 4) * 4, wave_max(gm));
  }
  asm volatile("s_waitcnt lgkmcnt(0)\n\ts_barrier" ::: "memory");
  if (wid == 0) STAMP(4);
  u32x4 o4[2];
  {
    const u32x4 ga = *(const u32x4*)(lds + C_GMAX);
    const u32 ab0 = *(const u32*)(lds + C_CTL) & 1;
    const float amax = __uint_as_float(max(max(ga[0], ga[1]), max(ga[2], ga[3])));
    const bool ab = ab0 || amax != amax;
    const float pre = amax < 0x1p-90f ? 0x1p100f : 1.0f;
    const float rinv = amax > 0.f ? 448.0f / (amax * pre) : 0.f;
    if (pre != 1.0f) {
#pragma unroll
      for (int i = 0; i < 8; ++i) z[i >> 2][i & 3] *= pre;
    }
#pragma unroll
    for (int t = 0; t < 2; ++t) {
#pragma unroll
      for (int e = 0; e < 4; ++e) {
        const f32x4 q = z[t][e] * rinv;
        o4[t][e] = (u32)__builtin_amdgcn_cvt_pk_fp8_f32(q[2], q[3], __builtin_amdgcn_cvt_pk_fp8_f32(q[0], q[1], 0, false), true);
      }
      if (ab) o4[t] = (u32x4){0x7f7f7f7fu, 0x7f7f7f7fu, 0x7f7f7f7fu, 0x7f7f7f7fu};
    }
  }
  if (wid == 0) STAMP(1);
  {
    const u32 pq = (wid + b + (b >> 3)) & 7;
    const u32 so = row0_of(rank, 2 * UB, 1) * 4096 + col0_of(rank, vB), vo = (16 * wid + r8) * 4096 + c * 16;
    if ((vB & 7) < 8 - DIRN) {
      const rsrc_t rm = mk_rsrc(p.out[rank], 16 * MiB);
      st128(o4[0], rm, vo, so);
      st128(o4[1], rm, vo, so + 32768);
      asm volatile("s_waitcnt vmcnt(0)\n\ts_barrier" ::: "memory");
      if (threadIdx.x == 0) __hip_atomic_store((u32*)(myws + OFF_FT) + tileB, seq, __ATOMIC_RELAXED, __HIP_MEMORY_SCOPE_SYSTEM);
      return;
    }
    *(u32x4*)(lds + 65536 + (16 * wid) * 128 + lane * 16) = o4[0];
    *(u32x4*)(lds + 65536 + (16 * wid + 8) * 128 + lane * 16) = o4[1];
    __syncthreads();
    const rsrc_t ro = mk_rsrc(p.out[pq], 16 * MiB);
#pragma unroll
    for (int t = 0; t < 16; ++t) {
      const u32 g = (t + 2 * pq + b) & 15;
      st128(*(const u32x4*)(lds + 65536 + g * 1024 + lane * 16), ro, r8 * 4096 + c * 16, so + g * 32768);
    }
    asm volatile("s_waitcnt vmcnt(0)" ::: "memory");
    if (wid == 0) STAMP(5);
    if (pq != rank) {
      if (lane == 0)
        __hip_atomic_store((u32*)(p.ws[pq] + OFF_FD + tileB * 64) + rank, seq, __ATOMIC_RELAXED, __HIP_MEMORY_SCOPE_SYSTEM);
      const u32* fd = (const u32*)(myws + OFF_FD + tileB * 64) + pq;
      while (__hip_atomic_load(fd, __ATOMIC_RELAXED, __HIP_MEMORY_SCOPE_SYSTEM) < seq) {
        if (__builtin_amdgcn_s_memrealtime() - t0 > DEADLINE) {
          if (lane == 0) p.out[rank][so] = 0x7f;
          break;
        }
        __builtin_amdgcn_s_sleep(8);
      }
    }
  }
}

extern "C" void kernel_launch(int rank, int nranks, ncclComm_t comm, hipStream_t stream, void* const* d_in,
                              const int* in_sizes, int n_in, void* d_out, int out_size, const int* in_dtypes,
                              int out_dtype, void* const* d_out_all, void* const* d_ws_all, size_t ws_bytes) {
  if (nranks != 8 || ws_bytes < WS_USED || (unsigned)rank >= 8u) {
    fprintf(stderr, "unsupported configuration\n");
    return;
  }
  (void)hipFuncSetAttribute((const void*)gemm_ar_q8_fused, hipFuncAttributeMaxDynamicSharedMemorySize, LDS_BYTES);
  Params p;
  p.x = d_in[0];
  p.w = d_in[1];
  for (int i = 0; i < 8; ++i) {
    p.ws[i] = (char*)d_ws_all[i];
    p.out[i] = (unsigned char*)d_out_all[i];
  }
  p.rank = rank;
  gemm_ar_q8_fused<<<256, 512, LDS_BYTES, stream>>>(p);
}
```
